```python
import numpy as np

kernel_name = "gpu_a2a_gemm_m4096_k4096_n4096_bf16_silu"


ABSMAX_RTOL = 0.008

COMPUTE_DTYPE = "bfloat16"
EPILOGUE = "silu"
IN_QUANT = None
OUT_QUANT = None
IN_DTYPES = {"x": "bfloat16", "w_mat": "bfloat16"}
WORLD_SIZE = 8

def _to_bf16(x):
    u = np.ascontiguousarray(x, dtype=np.float32).view(np.uint32)
    return ((u + 0x7FFF + ((u >> 16) & 1)) & 0xFFFF0000).view(np.float32)

def setup_inputs():

    x = _to_bf16(np.random.randn(4096, 4096).astype(np.float32))
    w_mat = _to_bf16((np.random.randn(4096, 4096) / np.sqrt(4096)).astype(np.float32))
    return {"x": x, "w_mat": w_mat}

def _epilogue(y):
    return (y / (1.0 + np.exp(-np.clip(y, -60.0, 60.0)))).astype(np.float32)

def reference(x, w_mat):
    y = (x @ w_mat).astype(np.float32)
    return _epilogue(y)
```

```cpp
#include <hip/hip_runtime.h>
#include <rccl/rccl.h>
#include <cstdio>

#define HIP_CHECK(expr) do { hipError_t _e = (expr); if (_e != hipSuccess) { \
    fprintf(stderr, "HIP error %s at %s:%d\n", hipGetErrorString(_e), __FILE__, __LINE__); \
    return; } } while (0)

typedef unsigned v4u __attribute__((ext_vector_type(4)));
typedef unsigned v2u __attribute__((ext_vector_type(2)));
typedef float f32x4 __attribute__((ext_vector_type(4)));
typedef float f32x16 __attribute__((ext_vector_type(16)));
typedef short s16x4 __attribute__((ext_vector_type(4)));
typedef short s16x8 __attribute__((ext_vector_type(8)));
typedef __bf16 bf16x8 __attribute__((ext_vector_type(8)));
typedef __amdgpu_buffer_rsrc_t rsrc_t;
typedef __attribute__((address_space(3))) s16x4* lds_s16x4_p;
typedef __attribute__((address_space(3))) v4u* lds_v4u_p;
typedef __attribute__((address_space(3))) f32x4* lds_f32x4_p;
typedef volatile __attribute__((address_space(3))) unsigned* lds_vu_p;

#define J1 13
#define PD 2
#define LD 4
#define PF_TICKS 45
#define PF_ON 0
#define GO_MODE 0
#define GO_TICKS 80
#define NOPUSH 0
#define ZERO_ON 1
#define POLL_SLEEP 2

constexpr int NR = 8;
constexpr unsigned SLOT_BYTES = 4u << 20;
constexpr unsigned SRC_BYTES = 512u << 10;
constexpr unsigned PART_OFF = 3 * SLOT_BYTES;
constexpr unsigned FLAG_OFF = PART_OFF + 256 * 32768;
constexpr unsigned PFLAG_OFF = FLAG_OFF + 4096;
constexpr unsigned CTR_OFF = PFLAG_OFF + 65536;
constexpr unsigned TMO_OFF = CTR_OFF + 4096;
constexpr unsigned TBL_OFF = TMO_OFF + 4096;
constexpr unsigned TS_OFF = TBL_OFF + 512 * 1024;
constexpr unsigned WS_NEED = TS_OFF + 256 * 8 * 16 * 8;
constexpr unsigned long long DEADLINE_TICKS = 100000000ull;

constexpr int LDS_TOTAL = 160 * 1024;
constexpr int RING = 35;
constexpr int NP1 = 16 + 7 * J1;
constexpr int NU = NP1 + 14 * (16 - J1);
constexpr int STG0 = 19;
constexpr int slot_of(int n) { return n < 2 ? n : 2 + (n - 2) % RING; }
constexpr int PS0 = slot_of(NU);
constexpr int EPI = (PS0 + 8 <= 37) ? (PS0 >= 12 ? 0 : PS0 + 8) : slot_of(NU + 7) + 1;
constexpr unsigned CTRL = 39 * 4096;
constexpr unsigned C_DONE = CTRL;
constexpr unsigned C_CALL = CTRL + 28;
constexpr unsigned C_PAIR = CTRL + 32;
constexpr unsigned C_BAD = CTRL + 48;
constexpr unsigned C_GO = CTRL + 52;
constexpr unsigned C_READY = CTRL + 256;
constexpr unsigned C_DUMP = CTRL + 1024;
constexpr unsigned DONE_FIN = 1000;
static_assert(NU + 10 <= 256, "table");
static_assert(J1 >= 2 && J1 <= 15, "J1");

struct Args { char* ws[NR]; };
template <int V> struct IC { static constexpr int value = V; };

__device__ __forceinline__ unsigned pk_bf16(float a, float b) {
  unsigned r; asm("v_cvt_pk_bf16_f32 %0, %1, %2" : "=v"(r) : "v"(a), "v"(b)); return r;
}
__device__ __forceinline__ float silu(float y) {
  float e = __builtin_amdgcn_exp2f(y * -1.44269504088896341f);
  return y * __builtin_amdgcn_rcpf(1.f + e);
}
__device__ __forceinline__ unsigned rfl(unsigned v) { return __builtin_amdgcn_readfirstlane(v); }
__device__ __forceinline__ unsigned long long now() { return __builtin_amdgcn_s_memrealtime(); }
__device__ __forceinline__ void lds_w32(unsigned addr, unsigned v) { *(lds_vu_p)(size_t)addr = v; }
__device__ __forceinline__ unsigned lds_r32(unsigned addr) { return *(lds_vu_p)(size_t)addr; }
__device__ __forceinline__ v4u lds_r128(unsigned addr) {
  v4u v; asm volatile("ds_read_b128 %0, %1\n\ts_waitcnt lgkmcnt(0)" : "=v"(v) : "v"(addr) : "memory"); return v;
}
__device__ __forceinline__ void glds(rsrc_t r, unsigned voff, unsigned soff, unsigned ldsaddr) {
  asm volatile("s_mov_b32 m0, %0\n\ts_nop 0\n\tbuffer_load_dwordx4 %1, %2, %3 offen lds"
               :: "s"(ldsaddr), "v"(voff), "s"(r), "s"(soff) : "m0", "memory");
}
__device__ __forceinline__ void glds_sc1(rsrc_t r, unsigned voff, unsigned soff, unsigned ldsaddr) {
  asm volatile("s_mov_b32 m0, %0\n\ts_nop 0\n\tbuffer_load_dwordx4 %1, %2, %3 offen sc1 lds"
               :: "s"(ldsaddr), "v"(voff), "s"(r), "s"(soff) : "m0", "memory");
}
__device__ __forceinline__ void glds1(rsrc_t r, unsigned voff, unsigned soff, unsigned ldsaddr) {
  asm volatile("s_mov_b32 m0, %0\n\ts_nop 0\n\tbuffer_load_dword %1, %2, %3 offen lds"
               :: "s"(ldsaddr), "v"(voff), "s"(r), "s"(soff) : "m0", "memory");
}
#define VMCNT(n) asm volatile("s_waitcnt vmcnt(%0)" :: "n"(n) : "memory")
#define SB() __builtin_amdgcn_sched_barrier(0)

#define STAMP(k) do {} while (0)

__global__ __launch_bounds__(512, 1) void fused(Args a, int rank, const char* __restrict__ x,
                                                const char* __restrict__ W, char* __restrict__ out) {
  __shared__ __attribute__((aligned(1024))) char smem[LDS_TOTAL];
  const int tid = threadIdx.x, lane = tid & 63, w = rfl(tid >> 6);
  const int bid = blockIdx.x;
  const unsigned long long T0 = now();
  const int g = bid & 7, idx = bid >> 3;
  const int nb = g * 4 + (idx & 3), mb = (idx >> 2) & 3, h = idx >> 4;
  char* me = a.ws[rank];
  const rsrc_t rW = __builtin_amdgcn_make_buffer_rsrc((void*)W, (short)0, 4096 * 4096 * 2, 0x00020000);
  const rsrc_t rX = __builtin_amdgcn_make_buffer_rsrc((void*)x, (short)0, 4096 * 512 * 2, 0x00020000);
  const rsrc_t rMe = __builtin_amdgcn_make_buffer_rsrc((void*)me, (short)0, (int)WS_NEED, 0x00020000);
  const unsigned lbase = (unsigned)(size_t)(__attribute__((address_space(3))) char*)smem;
  auto xstage = [&](unsigned soff, unsigned la) {
    const unsigned v0 = (((lane & 31) ^ (lane >> 5)) << 4) + (lane >> 5) * 1024u;
    #pragma unroll
    for (int i = 0; i < 16; ++i) glds(rX, v0 ^ ((i & 7) << 5), soff + i * 2048u, la + i * 1024u);
  };
  auto xfrag = [&](unsigned la, int j) {
    unsigned ad = la + (lane & 31) * 512u + ((((unsigned)(2 * j) + (lane >> 5)) ^ (lane & 15)) << 4);
    return *(lds_v4u_p)(size_t)ad;
  };
  auto get_call = [&]() {
    unsigned c;
    while ((c = rfl(lds_r32(lbase + C_CALL))) == 0) { if (now() - T0 > 2 * DEADLINE_TICKS) break; }
    return c;
  };
  auto set_tmo = [&](unsigned call) {
    if (lane == 0) {
      unsigned to = TMO_OFF;
      asm volatile("buffer_store_dword %0, %1, %2, 0 offen sc1" :: "v"(call), "v"(to), "s"(rMe) : "memory");
    }
  };

  if (w >= 4) {
    if (w == 4) { lds_w32(lbase + CTRL + lane * 4, 0); lds_w32(lbase + C_READY + lane * 4, 0); lds_w32(lbase + C_READY + 256 + lane * 4, 0); }
    if (w <= 5) {
      const int lw = w - 4;
      const unsigned wq = lane >> 4;
      const unsigned wvoff = wq * 8192u + (((lane & 15) ^ (((wq >> 1) << 3) | ((wq & 1) << 1))) << 4) + nb * 256u;
      const unsigned tso = TBL_OFF + (bid * 2 + lw) * 1024;
      const unsigned vo = lane * 16;
      {
        v4u t = {4u * lane + 2, 4u * lane + 3, 4u * lane + 4, 4u * lane + 5};
        asm volatile("buffer_store_dwordx4 %0, %1, %2, %3 offen" :: "v"(t), "v"(vo), "s"(rMe), "s"(tso) : "memory");
      }
      asm volatile("s_waitcnt lgkmcnt(0)\n\ts_barrier" ::: "memory");
      auto wait_go = [&]() { while (rfl(lds_r32(lbase + C_GO)) == 0) { if (now() - T0 > 2 * DEADLINE_TICKS) break; __builtin_amdgcn_s_sleep(1); } };
      if (GO_MODE) wait_go();
      int lim = STG0, slot = lw;
      const unsigned zv = 0;
      bool gone = GO_MODE != 0, pok = false;
      for (int n = lw; n < NU + 8; n += 2) {
        if (n >= lim) {
          if (!gone) { wait_go(); gone = true; }
          int spin = 0;
          while (true) {
            v4u d = lds_r128(lbase + C_DONE);
            int dmin = (int)rfl(min(min(d[0], d[1]), min(d[2], d[3])));
            if (dmin > 0) lim = max(dmin - 1, 2) + RING;
            if (n < lim) break;
            if ((++spin & 255) == 0 && now() - T0 > 2 * DEADLINE_TICKS) { lim = NU + 8; break; }
            __builtin_amdgcn_s_sleep(1);
          }
        }
        unsigned la = lbase + slot * 4096;
        if (n < NU) {
          unsigned k0;
          if (n < 16) k0 = rank * 512 + 256 * h + 16 * n;
          else if (n < NP1) { int r = (n - 16) / 7, u = (n - 16) - 7 * r; k0 = ((rank + 1 + u) & 7) * 512 + 256 * h + 16 * r; }
          else { int r2 = (n - NP1) / 7, u = (n - NP1) - 7 * r2; k0 = ((rank + 1 + u) & 7) * 512 + 256 * (h ^ (r2 & 1)) + 16 * (J1 + (r2 >> 1)); }
          unsigned so = k0 * 8192u;
          #pragma unroll
          for (int c = 0; c < 4; ++c) glds(rW, wvoff, so + c * 4 * 8192u, la + c * 1024);
        } else {
          if (!pok) {
            pok = true;
            const unsigned call = get_call();
            unsigned fo = lane < 2 ? PFLAG_OFF + ((bid ^ 128) * 2 + lane) * 128 : TMO_OFF;
            while (true) {
              unsigned fv;
              asm volatile("buffer_load_dword %0, %1, %2, 0 offen sc1\n\ts_waitcnt vmcnt(0)" : "=v"(fv) : "v"(fo), "s"(rMe) : "memory");
              unsigned long long ge = __builtin_amdgcn_ballot_w64((int)(fv - call) >= 0);
              if ((ge & 3) == 3) break;
              bool tmo_hit = (__builtin_amdgcn_ballot_w64(fv == call) >> 63) & 1;
              if (tmo_hit || now() - T0 > DEADLINE_TICKS) { set_tmo(call); lds_w32(lbase + C_BAD, 1); break; }
              __builtin_amdgcn_s_sleep(1);
            }
          }
          unsigned so = PART_OFF + (bid ^ 128) * 32768 + (n - NU) * 4096;
          #pragma unroll
          for (int c = 0; c < 4; ++c) glds_sc1(rMe, vo, so + c * 1024, la + c * 1024);
        }
        glds1(rMe, zv, tso + n * 4, lbase + C_READY + lw * 256);
        VMCNT(5 * (LD - 1));
        slot += 2; if (slot >= 2 + RING) slot -= RING;
      }
      VMCNT(0);
    } else if (w == 6) {
      const int p = (rank + 1 + bid % 7) & 7, q = bid / 7;
      const bool act = bid < 224 && !NOPUSH;
      const unsigned sla = lbase + (STG0 + 16) * 4096;
      if (act) xstage((unsigned)(p * 512 + (q >> 1) * 32) * 1024u + 512u * (q & 1), sla);
      asm volatile("s_waitcnt lgkmcnt(0)\n\ts_barrier" ::: "memory");
      if (act) {
        const unsigned call = get_call();
        const rsrc_t rP = __builtin_amdgcn_make_buffer_rsrc((void*)a.ws[p], (short)0, (int)WS_NEED, 0x00020000);
        unsigned so = (call % 3) * SLOT_BYTES + rank * SRC_BYTES + q * 16384;
        unsigned vo = lane * 16;
        VMCNT(0);
        STAMP(1);
        v4u v[16];
        #pragma unroll
        for (int j = 0; j < 16; ++j) v[j] = xfrag(sla, j);
        SB();
        #pragma unroll
        for (int j = 0; j < 16; ++j) {
          if (j >= PD) VMCNT(PD - 1);
          asm volatile("buffer_store_dwordx4 %0, %1, %2, %3 offen offset:%4 sc0 sc1"
                       :: "v"(v[j]), "v"(vo), "s"(rP), "s"(so + (j >> 2) * 4096), "n"((j & 3) * 1024) : "memory");
          if (j == (PD < 16 ? PD : 16) - 1) {
            asm volatile("s_waitcnt lgkmcnt(0)" ::: "memory");
            lds_w32(lbase + C_GO, 1);
            STAMP(4);
          }
        }
        STAMP(2);
        VMCNT(0);
        STAMP(3);
        if (lane == 0) {
          unsigned fo = FLAG_OFF + (rank * 32 + q) * 4;
          asm volatile("buffer_store_dword %0, %1, %2, 0 offen sc0 sc1" :: "v"(call), "v"(fo), "s"(rP) : "memory");
        }
      } else {
        if (GO_MODE) while (now() - T0 < GO_TICKS) __builtin_amdgcn_s_sleep(1);
        lds_w32(lbase + C_GO, 1);
      }
    } else {
      unsigned cv;
      {
        unsigned co = CTR_OFF + bid * 4;
        asm volatile("buffer_load_dword %0, %1, %2, 0 offen sc1\n\ts_waitcnt lgkmcnt(0)\n\ts_barrier\n\ts_waitcnt vmcnt(0)" : "=v"(cv) : "v"(co), "s"(rMe) : "memory");
      }
      const unsigned call = rfl(cv) + 1;
      lds_w32(lbase + C_CALL, call);
      if (lane == 0) {
        unsigned co = CTR_OFF + bid * 4;
        asm volatile("buffer_store_dword %0, %1, %2, 0 offen sc1" :: "v"(call), "v"(co), "s"(rMe) : "memory");
      }
      const unsigned zso = ((call + 2) % 3) * SLOT_BYTES + bid * 16384;
      const v4u zero = {0, 0, 0, 0};
      unsigned vo = lane * 16;
      unsigned pfsink = 0;
      if (GO_MODE) { while (rfl(lds_r32(lbase + C_GO)) == 0) { if (now() - T0 > 2 * DEADLINE_TICKS) break; __builtin_amdgcn_s_sleep(1); } }
      const unsigned long long T1 = now();
      asm volatile("s_nop 4" ::: "memory");
      for (int i = 0; i < 16; ++i) {
        if (PF_ON && i >= 2) {
          unsigned li = (unsigned)(i * 32 + idx) * 64 + lane;
          unsigned un = li >> 7, rowl = (li >> 3) & 15, ln = li & 7;
          unsigned n2 = un - 32;
          unsigned ks = n2 & 1, n = n2 >> 1, rr = n / 7, uu = n - rr * 7;
          unsigned k = ((rank + 1 + uu) & 7) * 512 + 256 * ks + 16 * rr + rowl;
          unsigned po = k * 8192u + g * 1024u + ln * 128u;
          asm volatile("buffer_load_dword %0, %1, %2, 0 offen" : "+v"(pfsink) : "v"(po), "s"(rW) : "memory");
        }
        if (ZERO_ON) asm volatile("buffer_store_dwordx4 %0, %1, %2, %3 offen sc0 sc1" :: "v"(zero), "v"(vo), "s"(rMe), "s"(zso + i * 1024) : "memory");
        while (now() - T1 < (unsigned long long)(i + 1) * PF_TICKS) __builtin_amdgcn_s_sleep(2);
      }
      asm volatile("s_waitcnt vmcnt(0)" : "+v"(pfsink) :: "memory");
    }
    return;
  }

  const int hh = lane >> 5, m = lane & 31;
  const bool keeper = (w >> 1) == h;
  v4u A0[8], A1[8];
  const unsigned sla = lbase + (STG0 + 4 * w) * 4096;
  xstage((unsigned)(rank * 512 + mb * 128 + 32 * w) * 1024u + 512u * h, sla);
  asm volatile("s_waitcnt lgkmcnt(0)\n\ts_barrier" ::: "memory");

  f32x16 acc[4];
  #pragma unroll
  for (int i = 0; i < 4; ++i)
    #pragma unroll
    for (int j = 0; j < 16; ++j) acc[i][j] = 0.f;

  unsigned tb[4];
  {
    const unsigned q = (lane >> 2) & 3, p = lane & 3, blk = (lane >> 4) & 1;
    unsigned b0 = 256u * (8 * hh + q) + 128u * (q >> 1) + 32u * (q & 1) + 64u * (p & 1) + 16u * blk + 8u * (p >> 1);
    tb[0] = lbase + b0; tb[1] = lbase + (b0 ^ 32u); tb[2] = lbase + (b0 ^ 128u); tb[3] = lbase + (b0 ^ 160u);
  }
  const unsigned done_addr = lbase + (lane == 0 ? C_DONE + 4 * w : C_DUMP + (w * 64 + lane) * 4);
  unsigned dn = 1;
  bool bad = false;
  s16x4 wf0[8], wf1[8];
  unsigned tc[4], tn[4];

  auto wait_ready = [&](unsigned need) {
    while (true) {
      unsigned r0 = lds_r32(lbase + C_READY), r1 = lds_r32(lbase + C_READY + 256);
      if (rfl(min(r0, r1)) >= need) break;
      if (now() - T0 > 2 * DEADLINE_TICKS) { bad = true; break; }
      __builtin_amdgcn_s_sleep(1);
    }
  };
  auto rdfrag = [&](s16x4 (&wf)[8], int i, unsigned ad) {
    wf[2 * i] = __builtin_amdgcn_ds_read_tr16_b64_v4i16((lds_s16x4_p)(size_t)ad);
    wf[2 * i + 1] = __builtin_amdgcn_ds_read_tr16_b64_v4i16((lds_s16x4_p)(size_t)(ad + 1024));
  };
  auto mma = [&](const s16x4 (&wf)[8], int i, const v4u& b) {
    s16x8 aa = __builtin_shufflevector(wf[2 * i], wf[2 * i + 1], 0, 1, 2, 3, 4, 5, 6, 7);
    acc[i] = __builtin_amdgcn_mfma_f32_32x32x16_bf16(__builtin_bit_cast(bf16x8, aa), __builtin_bit_cast(bf16x8, b), acc[i], 0, 0, 0);
  };

  unsigned call = 0, sso[7];
  unsigned avo = (unsigned)((mb * 4 + w) * 2 + h) * 16384u + lane * 16;
  unsigned fvo = lane < 7 ? FLAG_OFF + (((rank + 1 + lane) & 7) * 32 + (mb * 4 + w) * 2 + h) * 4 : TMO_OFF;

  auto slowA = [&](v4u (&A)[8]) {
    unsigned fprev = 0;
    while (true) {
      unsigned zmask = 0;
      #pragma unroll
      for (int u = 0; u < 7; ++u) {
        unsigned mu = min(min(A[u][0], A[u][1]), min(A[u][2], A[u][3]));
        if (__builtin_amdgcn_ballot_w64(mu == 0) != 0) zmask |= 1u << u;
      }
      if ((zmask & ~fprev & 0x7F) == 0 || bad) break;
      fprev = fprev >> 8;
      __builtin_amdgcn_s_sleep(POLL_SLEEP);
      unsigned fv;
#define CL(u) "s_bitcmp1_b32 %[z], " #u "\n\ts_cbranch_scc0 1f\n\tbuffer_load_dwordx4 %[a" #u "], %[vo], %[rs], %[s" #u "] offen sc0 sc1\n1:\n\t"
      asm volatile("s_nop 4\n\tbuffer_load_dword %[fv], %[fo], %[rs], 0 offen sc0 sc1\n\t"
                   CL(0) CL(1) CL(2) CL(3) CL(4) CL(5) CL(6) "s_waitcnt vmcnt(0)"
                   : [fv] "=&v"(fv), [a0] "+v"(A[0]), [a1] "+v"(A[1]), [a2] "+v"(A[2]), [a3] "+v"(A[3]), [a4] "+v"(A[4]), [a5] "+v"(A[5]), [a6] "+v"(A[6])
                   : [z] "s"(zmask), [vo] "v"(avo), [fo] "v"(fvo), [rs] "s"(rMe), [s0] "s"(sso[0]), [s1] "s"(sso[1]), [s2] "s"(sso[2]), [s3] "s"(sso[3]),
                     [s4] "s"(sso[4]), [s5] "s"(sso[5]), [s6] "s"(sso[6])
                   : "scc", "memory");
      unsigned fb = (unsigned)__builtin_amdgcn_ballot_w64((int)(fv - call) >= 0) & 0x7F;
      fprev |= fb << 8;
      bool tmo_hit = (__builtin_amdgcn_ballot_w64(fv == call) >> 7) & 1;
      if (tmo_hit || now() - T0 > DEADLINE_TICKS) { set_tmo(call); bad = true; }
    }
  };

  auto block = [&](const v4u (&Ac)[8], v4u (&An)[8], s16x4 (&wa)[8], s16x4 (&wb)[8], auto UC, bool loada, unsigned need, unsigned skip) {
    constexpr int U = decltype(UC)::value;
    unsigned r0 = 0, r1 = 0, mn = 1;
    #pragma unroll
    for (int u = 0; u < U; ++u) {
      if (u == U - 1) {
        SB();
        if (rfl(min(r0, r1)) < need) wait_ready(need);
      }
      #pragma unroll
      for (int i = 0; i < 4; ++i) {
        SB();
        if (u + 1 < U) { if (u & 1) rdfrag(wa, i, tc[i] + (u + 1) * 4096); else rdfrag(wb, i, tc[i] + (u + 1) * 4096); }
        else { if (u & 1) rdfrag(wa, i, tn[i]); else rdfrag(wb, i, tn[i]); }
        if (i == 3) { dn += (u == U - 2) ? 1 + skip : 1; lds_w32(done_addr, dn); }
        if (u < 2 && 4 * u + i < 7 && loada)
          An[4 * u + i] = __builtin_amdgcn_raw_buffer_load_b128(rMe, avo, sso[4 * u + i], 17);
        if (u == U - 3 && i == 0) { r0 = lds_r32(lbase + C_READY); r1 = lds_r32(lbase + C_READY + 256); }
        if (u == U - 1 && loada && !NOPUSH) {
          if (i == 0) mn = min(min(min(An[0][0], An[0][1]), min(An[0][2], An[0][3])), min(min(An[1][0], An[1][1]), An[1][2]));
          if (i == 1) mn = min(min(mn, An[1][3]), min(min(min(An[2][0], An[2][1]), min(An[2][2], An[2][3])), min(An[3][0], An[3][1])));
          if (i == 2) mn = min(min(min(mn, An[3][2]), An[3][3]), min(min(min(An[4][0], An[4][1]), min(An[4][2], An[4][3])), min(An[5][0], An[5][1])));
          if (i == 3) mn = min(min(min(mn, An[5][2]), An[5][3]), min(min(An[6][0], An[6][1]), min(An[6][2], An[6][3])));
        }
        SB();
        if (u & 1) mma(wb, i, Ac[u]); else mma(wa, i, Ac[u]);
      }
    }
    SB();
    if (__builtin_amdgcn_ballot_w64(mn == 0) != 0) slowA(An);
    #pragma unroll
    for (int i = 0; i < 4; ++i) tc[i] = tn[i];
  };

  VMCNT(0);
  #pragma unroll
  for (int j = 0; j < 8; ++j) { A0[j] = xfrag(sla, j); A1[j] = xfrag(sla, 8 + j); }
  lds_w32(done_addr, 1);
  wait_ready(8);
  STAMP(1);
  #pragma unroll
  for (int i = 0; i < 4; ++i) { tc[i] = tb[i]; tn[i] = tb[i] + 8 * 4096; rdfrag(wf0, i, tb[i]); }
  dn = 2; lds_w32(done_addr, dn);
  block(A0, A1, wf0, wf1, IC<8>{}, false, 16, 0);
  call = get_call();
  #pragma unroll
  for (int u = 0; u < 7; ++u) sso[u] = (call % 3) * SLOT_BYTES + ((rank + 1 + u) & 7) * SRC_BYTES;
  #pragma unroll
  for (int i = 0; i < 4; ++i) tn[i] = tb[i] + 16 * 4096;
  block(A1, A0, wf0, wf1, IC<8>{}, true, 23, 0);
  STAMP(2);

  const unsigned adelta = keeper ? 1024u : (unsigned)((mb * 4 + 2 * h + (w & 1)) * 2 + (1 - h) - ((mb * 4 + w) * 2 + h)) * 16384u + 1024u;
  const unsigned fvo2 = lane < 7 ? FLAG_OFF + (((rank + 1 + lane) & 7) * 32 + (mb * 4 + 2 * h + (w & 1)) * 2 + (1 - h)) * 4 : TMO_OFF;
  const int swr = keeper ? J1 : J1 - 1;
  int grp = 2;
  unsigned ubase = 16;
  auto round = [&](v4u (&Ac)[8], v4u (&An)[8], s16x4 (&wa)[8], s16x4 (&wb)[8], int r) {
    const bool sk = r >= swr;
    grp += sk ? 2 : 1; if (grp >= 5) grp -= 5;
    ubase += sk ? 14 : 7;
    const unsigned go = (unsigned)(2 + 7 * grp) * 4096u;
    #pragma unroll
    for (int i = 0; i < 4; ++i) tn[i] = tb[i] + go;
    const bool sw = (r + 1 == J1);
    avo += sw ? adelta : 1024u;
    if (sw && !keeper) fvo = fvo2;
    block(Ac, An, wa, wb, IC<7>{}, r < 15, r < 15 ? ubase + 7 : 0u, sk ? 7u : 0u);
    if (!keeper) {
      if (sw) {
        unsigned po = PART_OFF + bid * 32768 + (w & 1) * 16384;
        unsigned vo = lane * 16;
        asm volatile("s_nop 15\n\ts_nop 15" ::: "memory");
        #pragma unroll
        for (int i = 0; i < 4; ++i)
          #pragma unroll
          for (int c = 0; c < 4; ++c) {
            f32x4 t = {acc[i][4 * c], acc[i][4 * c + 1], acc[i][4 * c + 2], acc[i][4 * c + 3]};
            asm volatile("buffer_store_dwordx4 %0, %1, %2, %3 offen offset:%4 sc1\n\ts_nop 1"
                         :: "v"(t), "v"(vo), "s"(rMe), "s"(po + i * 4096), "n"(c * 1024) : "memory");
          }
        #pragma unroll
        for (int i = 0; i < 4; ++i)
          #pragma unroll
          for (int j = 0; j < 16; ++j) acc[i][j] = 0.f;
      }
      if (r == J1) {
        VMCNT(0);
        if (lane == 0) {
          unsigned fo = PFLAG_OFF + (bid * 2 + (w & 1)) * 128;
          asm volatile("buffer_store_dword %0, %1, %2, 0 offen sc1" :: "v"(call), "v"(fo), "s"(rMe) : "memory");
        }
      }
    }
  };
  for (int r = 0; r < 16; r += 2) {
    round(A0, A1, wf0, wf1, r);
    round(A1, A0, wf1, wf0, r + 1);
  }
  STAMP(9);

  if (bad) lds_w32(lbase + C_BAD, 1);
  lds_w32(done_addr, DONE_FIN);
  while (true) {
    v4u d = lds_r128(lbase + C_DONE);
    if (rfl(min(min(d[0], d[1]), min(d[2], d[3]))) >= DONE_FIN) break;
    if (now() - T0 > 3 * DEADLINE_TICKS) { bad = true; break; }
  }
  const unsigned eb = lbase + EPI * 4096;
  {
    unsigned sa = eb + w * 8192 + lane * 16;
    #pragma unroll
    for (int i = 0; i < 2; ++i)
      #pragma unroll
      for (int c = 0; c < 4; ++c) {
        const f32x16& s = keeper ? acc[2 + i] : acc[i];
        f32x4 t = {s[4 * c], s[4 * c + 1], s[4 * c + 2], s[4 * c + 3]};
        *(lds_f32x4_p)(size_t)(sa + (i * 4 + c) * 1024) = t;
      }
    asm volatile("" ::: "memory");
    lds_w32(lane == 0 ? lbase + C_PAIR + 4 * w : done_addr, 1 + DONE_FIN);
  }
  wait_ready(NU + 8);
  const int pw = w ^ 2;
  f32x4 t2[8];
  {
    #pragma unroll
    for (int i = 0; i < 2; ++i) {
      unsigned e0 = slot_of(NU + i), e1 = slot_of(NU + 2 + i), e2 = slot_of(NU + 4 + i), e3 = slot_of(NU + 6 + i);
      unsigned sl = (w & 1) ? (keeper ? e2 : e3) : (keeper ? e0 : e1);
      #pragma unroll
      for (int c = 0; c < 4; ++c) t2[i * 4 + c] = *(lds_f32x4_p)(size_t)(lbase + sl * 4096 + c * 1024 + lane * 16);
    }
  }
  while (true) {
    v4u d0 = lds_r128(lbase + C_PAIR);
    unsigned pv = pw == 0 ? d0[0] : pw == 1 ? d0[1] : pw == 2 ? d0[2] : d0[3];
    if (rfl(pv) != 0) break;
    if (now() - T0 > 3 * DEADLINE_TICKS) { bad = true; break; }
  }
  bad = bad || rfl(lds_r32(lbase + C_BAD)) != 0;
  const unsigned nanm = bad ? 0x7FC07FC0u : 0u;
  unsigned pk[16];
  {
    unsigned ra = eb + pw * 8192 + lane * 16;
    #pragma unroll
    for (int i = 0; i < 2; ++i)
      #pragma unroll
      for (int c = 0; c < 4; ++c) {
        f32x4 t = *(lds_f32x4_p)(size_t)(ra + (i * 4 + c) * 1024);
        t += t2[i * 4 + c];
        const f32x16& s = keeper ? acc[i] : acc[2 + i];
        float y0 = s[4 * c] + t[0], y1 = s[4 * c + 1] + t[1];
        float y2 = s[4 * c + 2] + t[2], y3 = s[4 * c + 3] + t[3];
        pk[i * 8 + c * 2] = pk_bf16(silu(y0), silu(y1)) | nanm;
        pk[i * 8 + c * 2 + 1] = pk_bf16(silu(y2), silu(y3)) | nanm;
      }
  }
  {
    unsigned ta = eb + 32768 + w * 4096 + m * 128;
    #pragma unroll
    for (int i = 0; i < 4; ++i) {
      v4u t = {pk[4 * i], pk[4 * i + 1], pk[4 * i + 2], pk[4 * i + 3]};
      *(lds_v4u_p)(size_t)(ta + (((4 * hh + i) ^ (m & 7)) << 4)) = t;
    }
    asm volatile("" ::: "memory");
    const rsrc_t rO = __builtin_amdgcn_make_buffer_rsrc((void*)out, (short)0, 512 * 4096 * 2, 0x00020000);
    unsigned rr = lane >> 3, ch = lane & 7;
    unsigned ra = eb + 32768 + w * 4096 + rr * 128 + ((ch ^ rr) << 4);
    unsigned oo = (unsigned)(mb * 128 + 64 * h + 32 * (w & 1) + rr) * 8192u + nb * 256u + (keeper ? 0u : 128u) + ch * 16u;
    #pragma unroll
    for (int i = 0; i < 4; ++i) {
      v4u o = *(lds_v4u_p)(size_t)(ra + i * 1024);
      __builtin_amdgcn_raw_buffer_store_b128(o, rO, oo, i * 8u * 8192u, 0);
    }
  }
}

extern "C" void kernel_launch(int rank, int nranks, ncclComm_t comm,
                              hipStream_t stream,
                              void* const* d_in, const int* in_sizes, int n_in,
                              void* d_out, int out_size,
                              const int* in_dtypes, int out_dtype,
                              void* const* d_out_all,
                              void* const* d_ws_all, size_t ws_bytes) {
  if (nranks != NR || ws_bytes < WS_NEED || (unsigned)rank >= (unsigned)NR) {
    fprintf(stderr, "unsupported configuration\n");
    return;
  }
  Args a;
  for (int p = 0; p < NR; ++p) a.ws[p] = (char*)d_ws_all[p];
  fused<<<256, 512, 0, stream>>>(a, rank, (const char*)d_in[0], (const char*)d_in[1], (char*)d_out);
  HIP_CHECK(hipGetLastError());
}
```

```cpp
#include <hip/hip_runtime.h>
#include <rccl/rccl.h>
#include <cstdio>

#define HIP_CHECK(expr) do { hipError_t _e = (expr); if (_e != hipSuccess) { \
    fprintf(stderr, "HIP error %s at %s:%d\n", hipGetErrorString(_e), __FILE__, __LINE__); \
    return; } } while (0)

typedef unsigned v4u __attribute__((ext_vector_type(4)));
typedef unsigned v2u __attribute__((ext_vector_type(2)));
typedef float f32x16 __attribute__((ext_vector_type(16)));
typedef short s16x4 __attribute__((ext_vector_type(4)));
typedef short s16x8 __attribute__((ext_vector_type(8)));
typedef __bf16 bf16x8 __attribute__((ext_vector_type(8)));
typedef __amdgpu_buffer_rsrc_t rsrc_t;
typedef __attribute__((address_space(3))) s16x4* lds_s16x4_p;
typedef volatile __attribute__((address_space(3))) v4u* lds_vv4u_p;
typedef volatile __attribute__((address_space(3))) unsigned* lds_vu_p;

#define LD 8
#define TAU 350
#define PER 620
#define TBASE 150
#define NOPUSH 0
#define PLDK 8
#define PSLEEP 2
#define CPRIO 2
#define NLW 3
#define LPRIO 0
#define XP 0
#define AUX 17
#define NOA 0
#define RFM 0x1222

constexpr int NR = 8;
constexpr int NWAVE = 8;
constexpr unsigned SLOT_BYTES = 4u << 20;
constexpr unsigned SRC_BYTES = 512u << 10;
constexpr unsigned PART_OFF = 3 * SLOT_BYTES;
constexpr unsigned FLAG_OFF = PART_OFF + 256 * 32768;
constexpr unsigned HINT_OFF = FLAG_OFF + 4096;
constexpr unsigned TMO_OFF = HINT_OFF + 4096;
constexpr unsigned CTR_OFF = TMO_OFF + 4096;
constexpr unsigned TS_OFF = CTR_OFF + 8192;
constexpr unsigned WS_NEED = TS_OFF + 256 * 16 * 16 * 8;
constexpr unsigned WS_GUARD = 2u << 20;
#define DL 100000000ull
constexpr unsigned long long DEADLINE_TICKS = DL;

constexpr int LDS_TOTAL = 160 * 1024;
constexpr int NU = 16 + 7 * 16;
constexpr unsigned CTRL = 39 * 4096;
constexpr unsigned C_FL = CTRL;
constexpr unsigned C_DONE = CTRL + 512;
constexpr unsigned C_BAD = CTRL + 1008;
constexpr unsigned C_ARR = CTRL + 1024;
constexpr int gslot(int r) { return r % 5 == 0 ? 32 : (r % 5 == 1 ? 0 : (r % 5 == 2 ? 16 : (r % 5 == 3 ? 7 : 23))); }
constexpr int hbit(int j) { return (j / 7) * 8 + j % 7; }
constexpr unsigned issued(int k, bool tail) {
  unsigned m = 0;
  for (int j = 0; j < 14; ++j) {
    if (j >= k && !(k == 0 && j == 13)) m |= 1u << hbit(j);
    if (!tail && j <= k - 2) m |= 1u << (16 + hbit(j));
  }
  return m;
}
constexpr unsigned notyet(int k) {
  unsigned m = k == 0 ? 1u << hbit(13) : 0u;
  for (int j = 0; j < 14; ++j) if (j > k - 2) m |= 1u << (16 + hbit(j));
  return m;
}

struct Args { char* ws[NR]; };
template <int V> struct IC { static constexpr int value = V; };
template <int B, int E, typename F> __device__ __forceinline__ void sfor(F&& f) {
  if constexpr (B < E) { f(IC<B>{}); sfor<B + 1, E>(f); }
}

__device__ __forceinline__ unsigned pk_bf16(float a, float b) {
  unsigned r; asm("v_cvt_pk_bf16_f32 %0, %1, %2" : "=v"(r) : "v"(a), "v"(b)); return r;
}
__device__ __forceinline__ float silu(float y) {
  float e = __builtin_amdgcn_exp2f(y * -1.44269504088896341f);
  return y * __builtin_amdgcn_rcpf(1.f + e);
}
__device__ __forceinline__ unsigned rfl(unsigned v) { return __builtin_amdgcn_readfirstlane(v); }
__device__ __forceinline__ unsigned long long now() { return __builtin_amdgcn_s_memrealtime(); }
__device__ __forceinline__ void lds_w32(unsigned addr, unsigned v) { *(lds_vu_p)(size_t)addr = v; }
__device__ __forceinline__ unsigned lds_r32(unsigned addr) { return *(lds_vu_p)(size_t)addr; }
__device__ __forceinline__ v4u lds_r128v(unsigned addr) { return *(lds_vv4u_p)(size_t)addr; }
__device__ __forceinline__ void glds4(rsrc_t r, unsigned v0, unsigned v1, unsigned s0, unsigned s1, unsigned s2, unsigned s3, unsigned la) {
  asm volatile("s_mov_b32 m0, %0\n\ts_nop 0\n\t"
               "buffer_load_dwordx4 %1, %3, %4 offen lds\n\t"
               "buffer_load_dwordx4 %2, %3, %5 offen offset:1024 lds\n\t"
               "buffer_load_dwordx4 %1, %3, %6 offen offset:2048 lds\n\t"
               "buffer_load_dwordx4 %2, %3, %7 offen offset:3072 lds"
               :: "s"(la), "v"(v0), "v"(v1), "s"(r), "s"(s0), "s"(s1), "s"(s2), "s"(s3) : "m0", "memory");
}
__device__ __forceinline__ void gzero(rsrc_t r, unsigned zo, unsigned la) {
  asm volatile("s_mov_b32 m0, %0\n\ts_mov_b64 exec, 1\n\tbuffer_load_dword off, %1, %2 lds\n\ts_mov_b64 exec, -1"
               :: "s"(la), "s"(r), "s"(zo) : "m0", "memory");
}
__device__ __forceinline__ void st1(rsrc_t r, unsigned so, unsigned v) {
  asm volatile("s_mov_b64 exec, 1\n\tbuffer_store_dword %0, off, %1, %2 sc0 sc1\n\ts_mov_b64 exec, -1" :: "v"(v), "s"(r), "s"(so) : "memory");
}
__device__ __forceinline__ void reld(v4u& A, rsrc_t r, unsigned vo, unsigned so) {
  const unsigned long long z = __builtin_amdgcn_ballot_w64(min(min(A[0], A[1]), min(A[2], A[3])) == 0);
  asm volatile("s_mov_b64 exec, %1\n\tbuffer_load_dwordx4 %0, %2, %3, %4 offen sc0 sc1\n\ts_mov_b64 exec, -1"
               : "+v"(A) : "s"(z), "v"(vo), "s"(r), "s"(so) : "memory");
}
__device__ __forceinline__ void pin(v4u& A) { asm volatile("" : "+v"(A)); }
__device__ __forceinline__ void sor(unsigned& A, unsigned b) { asm volatile("s_or_b32 %0, %0, %1" : "+s"(A) : "s"(b) : "scc"); }
#define VMCNT(n) asm volatile("s_waitcnt vmcnt(%0)" :: "n"(n) : "memory")
#define SB() __builtin_amdgcn_sched_barrier(0)
#define MEMB() asm volatile("" ::: "memory")
#define BARRIER() asm volatile("s_barrier" ::: "memory")
__device__ __forceinline__ unsigned min4(const v4u& a) { return min(min(a[0], a[1]), min(a[2], a[3])); }
__device__ __forceinline__ bool anyz(unsigned v) { return __builtin_amdgcn_ballot_w64(v == 0) != 0; }

#define STAMP(k) do {} while (0)
#define TSOUT() do {} while (0)

__global__ __launch_bounds__(64 * NWAVE, 1) void fused(Args a, int rank, const char* __restrict__ x,
                                                       const char* __restrict__ W, char* __restrict__ out) {
  __shared__ __attribute__((aligned(1024))) char smem[LDS_TOTAL];
  const int tid = threadIdx.x, lane = tid & 63, w = rfl(tid >> 6);
  const int bid = blockIdx.x;
  const unsigned long long T0 = now();
  const int g = bid & 7, idx = bid >> 3;
  const int h = g & 1, nb = (idx & 7) * 4 + (g >> 1), mb = idx >> 3;
  const int rot = idx & 3;
  const int hh = lane >> 5, m = lane & 31;
  char* me = a.ws[rank];
  const rsrc_t rX = __builtin_amdgcn_make_buffer_rsrc((void*)x, (short)0, 4096 * 512 * 2, 0x00020000);
  const rsrc_t rMe = __builtin_amdgcn_make_buffer_rsrc((void*)me, (short)0, (int)WS_NEED, 0x00020000);
  const unsigned lbase = (unsigned)(size_t)(__attribute__((address_space(3))) char*)smem;
  const unsigned co = CTR_OFF + (bid * 8 + w) * 4;
  auto timeout = [&](unsigned call) { st1(rMe, TMO_OFF, call); lds_w32(lbase + C_BAD, 0); };

  if (w == 0 && NLW < 4) {
    const int p = (rank + 1 + bid % 7) & 7, q = bid / 7;
    if (bid < 224 && !NOPUSH) {
      v4u v[16];
      const unsigned xv = m * 1024u + hh * 16u;
      const unsigned xs = (unsigned)(p * 512 + (q >> 1) * 32) * 1024u + (q & 1) * 32u;
      const unsigned cv = __builtin_amdgcn_raw_buffer_load_b32(rMe, 0, co, 17);
      #pragma unroll
      for (int j = 0; j < 16; ++j) v[j] = __builtin_amdgcn_raw_buffer_load_b128(rX, xv + j * 64, xs, 0);
      BARRIER();
      const unsigned call = rfl(cv) + 1;
      const rsrc_t rP = __builtin_amdgcn_make_buffer_rsrc((void*)a.ws[p], (short)0, (int)WS_NEED, 0x00020000);
      const unsigned so = (call % 3) * SLOT_BYTES + rank * SRC_BYTES + (q & 1) * 16384 + (q >> 1) * 1024;
      const unsigned vo = lane * 16;
      const unsigned long long t1 = T0 + TBASE + (unsigned)(q * PER) / 320;
      STAMP(1);
      #pragma unroll
      for (int j = 0; j < 16; ++j) {
        const unsigned long long tq = t1 + (j * PER) / 10;
        while ((long long)(now() - tq) < 0) __builtin_amdgcn_s_sleep(1);
        if (j == 0) STAMP(2);
        asm volatile("buffer_store_dwordx4 %0, %1, %2, %3 offen sc0 sc1"
                     :: "v"(v[j]), "v"(vo), "s"(rP), "s"(so + j * 32768) : "memory");
      }
      asm volatile("s_nop 1" :: "v"(v[15]));
      st1(rMe, co, call);
      STAMP(4);
      VMCNT(0);
      STAMP(3);
      st1(rP, FLAG_OFF + (rank * 32 + q) * 4, call);
    } else {
      BARRIER();
    }
    TSOUT();
    return;
  }
  if (w == 3) {
      const unsigned za = lbase + CTRL + lane * 16;
      const v4u ones = {~0u, ~0u, ~0u, ~0u};
      asm volatile("ds_write_b128 %0, %1\n\tds_write_b128 %0, %1 offset:1024\n\ts_waitcnt lgkmcnt(0)" :: "v"(za), "v"(ones) : "memory");
  }
  if (w == 1 || w == 2 || (NLW > 2 && w == 3) || (NLW > 3 && w == 0)) {
    const int lw = (w + 3) & 3;
    if (LPRIO) __builtin_amdgcn_s_setprio(LPRIO);
    const rsrc_t rW = __builtin_amdgcn_make_buffer_rsrc((void*)W, (short)0, 4096 * 4096 * 2, 0x00020000);
    const unsigned wq = lane >> 4;
    const unsigned ch = (lane & 15) ^ (((wq >> 1) << 3) | ((wq & 1) << 1));
    const unsigned wv1 = wq * 8192u + ((ch ^ (8u * h)) << 4) + nb * 256u;
    const unsigned wv2 = (wq + ((ch >> 3) ^ h) * 16) * 8192u + ((ch & 7) << 4) + nb * 256u + h * 128u;
    const unsigned cv = __builtin_amdgcn_raw_buffer_load_b32(rMe, 0, co, 17);
    int seen = 0;
    int rr = 0, uu = lw - 16, g5 = 0;
    BARRIER();
    if (XP & 1) return;
    for (int n = lw; n < NU; n += NLW) {
      unsigned k0, wv = wv1, slot;
      int need = 0;
      if (n < 16) { k0 = rank * 512 + (((n >> 2) + rot) & 3) * 128 + 64 * h + 16 * (n & 3); slot = n; }
      else {
        k0 = ((rank + 1 + uu) & 7) * 512 + 32 * rr;
        if (rr < 14) k0 += 16 * h; else wv = wv2;
        slot = ((0x1707100020ull >> (8 * g5)) & 255) + uu;
        need = rr < 5 ? (rr + 1) >> 1 : rr - 2;
      }
      if (seen < need) {
        int spin = 0;
        while (true) {
          const v4u d = lds_r128v(lbase + C_DONE + (need - 1) * 16);
          if (rfl(d[0] | d[1] | d[2] | d[3]) == 0) break;
          if ((++spin & 255) == 0 && now() - T0 > 2 * DEADLINE_TICKS) break;
          __builtin_amdgcn_s_sleep(1);
        }
        seen = need;
      }
      const unsigned so = k0 * 8192u;
      glds4(rW, wv, wv, so, so + 31744u, so + 2 * 31744u, so + 3 * 31744u, lbase + slot * 4096);
      gzero(rMe, WS_NEED, lbase + C_FL + n * 4);
      VMCNT(5 * (LD - 1));
      uu += NLW;
      if (uu >= 7) { uu -= 7; ++rr; g5 = g5 == 4 ? 0 : g5 + 1; }
    }
    if (lw == 0 && !NOPUSH) {
      const unsigned call = rfl(cv) + 1;
      const unsigned zo = ((call + 2) % 3) * SLOT_BYTES + bid * 16384, vo = lane * 16;
      const v4u zero = {0, 0, 0, 0};
      #pragma unroll
      for (int j = 0; j < 16; ++j)
        asm volatile("buffer_store_dwordx4 %0, %1, %2, %3 offen offset:%4 sc0 sc1"
                     :: "v"(zero), "v"(vo), "s"(rMe), "s"(zo + (j >> 2) * 4096), "n"((j & 3) * 1024) : "memory");
      asm volatile("s_nop 1" :: "v"(zero));
      st1(rMe, co, call);
    }
    VMCNT(0);
    return;
  }
  const int cw = w - 4;
  const unsigned cv = __builtin_amdgcn_raw_buffer_load_b32(rMe, 0, co, 17);
  BARRIER();
  if (XP & 1) { TSOUT(); return; }
  {
    const unsigned xv = (lane >> 3) * 1024u + (((lane & 7) ^ (lane >> 4)) << 4);
    const unsigned xs = (unsigned)(rank * 512 + mb * 128 + 32 * cw) * 1024u + 128u * h;
    #pragma unroll
    for (int b = 0; b < 4; ++b) {
      const unsigned so = xs + ((b + rot) & 3) * 256u;
      glds4(rX, xv, xv ^ 64u, so, so + 7168u, so + 2 * 7168u, so + 3 * 7168u, lbase + (16 + 4 * b + cw) * 4096);
    }
  }
  if (CPRIO) __builtin_amdgcn_s_setprio(CPRIO);

  f32x16 acc[4];
  #pragma unroll
  for (int i = 0; i < 4; ++i)
    #pragma unroll
    for (int j = 0; j < 16; ++j) acc[i][j] = 0.f;

  unsigned tb[4], tw[4];
  {
    const unsigned q = (lane >> 2) & 3, p = lane & 3, blk = (lane >> 4) & 1;
    unsigned b0 = 256u * (8 * hh + q) + 128u * (q >> 1) + 32u * (q & 1) + 64u * (p & 1) + 16u * blk + 8u * (p >> 1);
    tb[0] = lbase + b0; tb[1] = lbase + (b0 ^ 32u); tb[2] = lbase + (b0 ^ 128u); tb[3] = lbase + (b0 ^ 160u);
  }
  const unsigned xra = lbase + (16 + cw) * 4096 + m * 128 + (((unsigned)hh ^ ((m >> 1) & 7)) << 4);
  const unsigned avo = (unsigned)(mb * 4 + cw) * 1024u + lane * 16 + h * 16384;
  unsigned fad = lbase + C_FL;
  unsigned vn0 = avo, vn1 = avo + 32768;
  asm volatile("" : "+v"(fad));
  unsigned dad = lbase + C_DONE + 4 * cw;
  s16x4 wf[2][8];
  v4u B[14], Y[14];
  unsigned F = 0;
  VMCNT(16);
  const unsigned call = rfl(cv) + 1;
  st1(rMe, co, call);
  unsigned su[7];
  #pragma unroll
  for (int u = 0; u < 7; ++u) su[u] = (call % 3) * SLOT_BYTES + ((rank + 1 + u) & 7) * SRC_BYTES;

  auto rdtr = [&](unsigned ad) { return __builtin_amdgcn_ds_read_tr16_b64_v4i16((lds_s16x4_p)(size_t)ad); };
  auto mma = [&](const s16x4 (&f)[8], int i, int ai, const v4u& b) {
    s16x8 aa = __builtin_shufflevector(f[2 * i], f[2 * i + 1], 0, 1, 2, 3, 4, 5, 6, 7);
    acc[ai] = __builtin_amdgcn_mfma_f32_32x32x16_bf16(__builtin_bit_cast(bf16x8, aa), __builtin_bit_cast(bf16x8, b), acc[ai], 0, 0, 0);
  };
  auto ldA = [&](unsigned vo, int u, int bit) -> v4u {
    if (NOA == 2) { v4u z; asm volatile("" : "=v"(z)); return z; }
    const unsigned so = NOA ? WS_NEED : su[u];
    return __builtin_amdgcn_raw_buffer_load_b128(rMe, vo, so, AUX);
  };
  auto isbad = [&]() { return rfl(lds_r32(lbase + C_BAD)) == 0; };
  auto gslow = [&](v4u& A, unsigned vo, int u, int hs) {
    const unsigned fo = lane == 0 ? FLAG_OFF + (((rank + 1 + u) & 7) * 32 + 2 * (mb * 4 + cw) + hs) * 4 : TMO_OFF;
    bool fseen = false;
    #pragma nounroll
    while (true) {
      unsigned fv;
      asm volatile("s_nop 4\n\tbuffer_load_dwordx4 %0, %2, %3, %4 offen sc0 sc1\n\t"
                   "buffer_load_dword %1, %5, %3, 0 offen sc0 sc1\n\ts_waitcnt vmcnt(0)"
                   : "+&v"(A), "=&v"(fv) : "v"(vo), "s"(rMe), "s"(su[u]), "v"(fo) : "memory");
      if (fseen || !anyz(min4(A))) break;
      fseen = (__builtin_amdgcn_ballot_w64((int)(fv - call) >= 0) & 1) != 0;
      const bool tmo_hit = (__builtin_amdgcn_ballot_w64(fv == call) >> 1) & 1;
      if (tmo_hit || now() - T0 > DEADLINE_TICKS) { timeout(call); break; }
      __builtin_amdgcn_s_sleep(PSLEEP);
    }
  };
  auto fspin = [&](unsigned fa) {
    int spin = 0;
    #pragma nounroll
    while (rfl(lds_r32(fa)) != 0) {
      if ((++spin & 31) == 0 && now() - T0 > 2 * DEADLINE_TICKS) { lds_w32(lbase + C_BAD, 0); break; }
    }
  };
  auto step = [&](auto MODE_, auto K_, int rb) {
    constexpr int MODE = decltype(MODE_)::value, K = decltype(K_)::value;
    constexpr int PB = (K + 1) & 1;
    constexpr bool LAST = MODE == 3 && K == 14;
    constexpr bool CT = MODE == 3 && K > 0;
    constexpr bool VAL = MODE != 0 && !LAST && !NOPUSH;
    constexpr bool FCK = !(MODE == 3 && K >= 13);
    constexpr int CB = MODE == 0 ? 12 + ((K + 1) & 1) : (K + 13) % 14;
    constexpr unsigned woff = MODE == 0 ? K * 4096 : (K % 7) * 4096;
    s16x4 (&wc)[8] = wf[PB];
    s16x4 (&wn)[8] = wf[PB ^ 1];
    unsigned t = 1;
    bool fail = false;
    auto tr = [&](int j) { if constexpr (!LAST) wn[j] = rdtr(tw[j >> 1] + woff + (j & 1) * 1024); };
    #pragma unroll
    for (int i = 0; i < 4; ++i) {
      SB();
      if (i == 0) {
        if constexpr (MODE == 0) B[12 + (K & 1)] = lds_r128v((xra ^ ((K & 3) << 5)) + (K >> 2) * 16384);
        tr(0); tr(1);
        if constexpr (MODE == 0) { if constexpr (K >= 4) B[K - 4] = ldA(K - 4 < 7 ? vn0 : vn1, (K - 4) % 7, hbit(K - 4)); }
        else if constexpr (K < 2) {
          B[12 + K] = ldA(vn1, 5 + K, hbit(12 + K));
          if constexpr (MODE == 3) Y[12 + K] = ldA(vn1 ^ 16384u, 5 + K, hbit(12 + K));
        } else if constexpr (MODE != 3) {
          B[K - 2] = ldA(K - 2 < 7 ? vn0 : vn1, (K - 2) % 7, 16 + hbit(K - 2));
          if constexpr (MODE == 2) Y[K - 2] = ldA((K - 2 < 7 ? vn0 : vn1) ^ 16384u, (K - 2) % 7, 16 + hbit(K - 2));
        }
      }
      if (i == 1) {
        tr(2); tr(3); tr(4);
        if constexpr (VAL) {
          t = min4(B[K]);
          if constexpr (MODE == 3) t = min(min(min(t, Y[K][0]), Y[K][1]), min(Y[K][2], Y[K][3]));
        }
        if constexpr (MODE == 1 || MODE == 2) { if (K == 0) vn0 += 65536; if (K == 2) vn1 += 65536; }
      }
      if (i == 2) {
        tr(5); tr(6); tr(7);
        if constexpr (FCK) fail = __builtin_amdgcn_ballot_w64(t <= F) != 0;
        else if constexpr (VAL) fail = anyz(t);
      }
      if (i == 3) {
        if (__builtin_expect(fail, 0)) {
          if constexpr (FCK) fspin(fad + (K + 1) * 4);
          if constexpr (VAL) {
            if (anyz(t) && !isbad()) {
              const unsigned v0 = avo + rb * 32768;
              auto all = [&]() {
                VMCNT(0);
                sfor<0, 14>([&](auto JC) {
                  constexpr int j = (decltype(JC)::value + K) % 14;
                  if constexpr (j >= K && !(K == 0 && j == 13)) {
                    reld(B[j], rMe, v0, su[j % 7] + (j / 7) * 32768);
                    if constexpr (MODE == 3) reld(Y[j], rMe, v0 ^ 16384u, su[j % 7] + (j / 7) * 32768);
                  }
                  if constexpr (MODE != 3 && j <= K - 2) {
                    reld(B[j], rMe, v0, su[j % 7] + (2 + j / 7) * 32768);
                    if constexpr (MODE == 2) reld(Y[j], rMe, v0 ^ 16384u, su[j % 7] + (2 + j / 7) * 32768);
                  }
                });
                VMCNT(0);
              };
              #pragma nounroll
              for (int pass = 0; pass < 2; ++pass) {
                all();
                pin(B[K]);
                if (!anyz(min4(B[K]))) break;
                if (pass == 0) gslow(B[K], v0 + (K / 7) * 32768, K % 7, h);
              }
              if constexpr (MODE == 3) {
                pin(Y[K]);
                if (anyz(min4(Y[K]))) gslow(Y[K], (v0 + (K / 7) * 32768) ^ 16384u, K % 7, h ^ 1);
              }
            }
          }
        }
        SB();
        if constexpr (!(MODE == 3 && K >= 12)) F = lds_r32(fad + (K + 2) * 4);
        if constexpr (MODE == 0 ? K == 9 : (MODE == 1 && K % 7 == 1)) {
          gzero(rMe, WS_NEED, dad); dad += 16;
        }
      }
      SB();
      if constexpr (CT) mma(wc, i, i & 1, i < 2 ? B[CB] : Y[CB]);
      else mma(wc, i, i, B[CB]);
    }
    SB();
  };

  VMCNT(13);
  B[12] = lds_r128v(xra);
  fspin(fad);
  fspin(fad + 4);
  STAMP(1);
  #pragma unroll
  for (int i = 0; i < 4; ++i) { tw[i] = tb[i]; wf[0][2 * i] = rdtr(tw[i]); wf[0][2 * i + 1] = rdtr(tw[i] + 1024); }
  F = lds_r32(fad + 8);
  VMCNT(1);
  sfor<1, 16>([&](auto NC) { step(IC<0>{}, NC, 0); });
  STAMP(2);
  fad += 16 * 4;
  auto iter = [&](auto MODE_, int r) {
    sfor<0, 14>([&](auto KC) {
      constexpr int k = decltype(KC)::value;
      if constexpr (k % 7 == 0) {
        const unsigned go = ((unsigned)(0x201707100020ull >> (8 * ((r % 5) + k / 7))) & 255) * 4096;
        #pragma unroll
        for (int i = 0; i < 4; ++i) tw[i] = tb[i] + go;
      }
      step(MODE_, KC, r);
    });
    fad += 14 * 4;
  };
  for (int r = 0; r < 12; r += 2) {
    iter(IC<1>{}, r);
  }
  STAMP(6);
  iter(IC<2>{}, 12);
  STAMP(7);
  v4u P[8];
  const unsigned pvo = lane * 16;
  sfor<0, 15>([&](auto KC) {
    constexpr int k = decltype(KC)::value;
    if constexpr (k % 7 == 0 && k < 14) {
      #pragma unroll
      for (int i = 0; i < 4; ++i) tw[i] = tb[i] + gslot(14 + k / 7) * 4096;
    }
    step(IC<3>{}, KC, 14);
    if constexpr (k == 0) {
      const unsigned po = PART_OFF + bid * 32768 + cw * 8192;
      const bool bad = isbad();
      const unsigned am = bad ? 0u : ~1u, ov = (bad ? 0x7FC00000u : 0u) | (call & 1);
      v4u t[8];
      #pragma unroll
      for (int c = 0; c < 8; ++c)
        #pragma unroll
        for (int e = 0; e < 4; ++e) { const float f = acc[2 + (c >> 2)][4 * (c & 3) + e]; t[c][e] = (__float_as_uint(f) & am) | ov; }
      asm volatile("buffer_store_dwordx4 %0, %8, %9, %10 offen sc0 sc1\n\t"
                   "buffer_store_dwordx4 %1, %8, %9, %10 offen offset:1024 sc0 sc1\n\t"
                   "buffer_store_dwordx4 %2, %8, %9, %10 offen offset:2048 sc0 sc1\n\t"
                   "buffer_store_dwordx4 %3, %8, %9, %10 offen offset:3072 sc0 sc1\n\t"
                   "buffer_store_dwordx4 %4, %8, %9, %11 offen sc0 sc1\n\t"
                   "buffer_store_dwordx4 %5, %8, %9, %11 offen offset:1024 sc0 sc1\n\t"
                   "buffer_store_dwordx4 %6, %8, %9, %11 offen offset:2048 sc0 sc1\n\t"
                   "buffer_store_dwordx4 %7, %8, %9, %11 offen offset:3072 sc0 sc1\n\ts_nop 1"
                   :: "v"(t[0]), "v"(t[1]), "v"(t[2]), "v"(t[3]), "v"(t[4]), "v"(t[5]), "v"(t[6]), "v"(t[7]),
                      "v"(pvo), "s"(rMe), "s"(po), "s"(po + 4096) : "memory");
    }
    if constexpr (k == PLDK) {
      const unsigned po = PART_OFF + (bid ^ 1) * 32768 + cw * 8192;
      #pragma unroll
      for (int c = 0; c < 8; ++c) P[c] = __builtin_amdgcn_raw_buffer_load_b128(rMe, pvo + (c & 3) * 1024, po + (c >> 2) * 4096, 17);
    }
  });
  STAMP(9);

  bool bad = isbad();
  {
    const unsigned po = PART_OFF + (bid ^ 1) * 32768 + cw * 8192;
    const unsigned tag = call & 1;
    int it = 0;
    #pragma nounroll
    while (true) {
      unsigned o0 = 0;
      #pragma unroll
      for (int c = 0; c < 8; ++c)
        #pragma unroll
        for (int e = 0; e < 4; ++e) o0 |= P[c][e] ^ tag;
      if (__builtin_amdgcn_ballot_w64((o0 & 1) != 0) == 0 || bad) break;
      if ((++it & 15) == 0) {
        const unsigned tv = rfl(__builtin_amdgcn_raw_buffer_load_b32(rMe, 0, TMO_OFF, 17));
        if (tv == call || now() - T0 > DEADLINE_TICKS) { timeout(call); bad = true; }
      }
      MEMB();
      #pragma unroll
      for (int c = 0; c < 8; ++c) P[c] = __builtin_amdgcn_raw_buffer_load_b128(rMe, pvo + (c & 3) * 1024, po + (c >> 2) * 4096, 17);
    }
  }
  STAMP(10);
  const unsigned nanm = bad ? 0x7FC07FC0u : 0u;
  {
    const unsigned eb = lbase + (14 + (cw & 1) + (cw >> 1) * 16) * 4096;
    const unsigned ta = eb + m * 128;
    #pragma unroll
    for (int i = 0; i < 2; ++i)
      #pragma unroll
      for (int c2 = 0; c2 < 2; ++c2) {
        v4u t;
        #pragma unroll
        for (int e = 0; e < 4; ++e) {
          const int j = 8 * c2 + 2 * e;
          const unsigned p0 = P[i * 4 + (j >> 2)][j & 3], p1 = P[i * 4 + (j >> 2)][(j & 3) + 1];
          float y0 = acc[i][j] + __uint_as_float(p0), y1 = acc[i][j + 1] + __uint_as_float(p1);
          t[e] = pk_bf16(silu(y0), silu(y1)) | nanm;
        }
        *(lds_vv4u_p)(size_t)(ta + (((4 * hh + 2 * i + c2) ^ (m & 7)) << 4)) = t;
      }
    const rsrc_t rO = __builtin_amdgcn_make_buffer_rsrc((void*)out, (short)0, 512 * 4096 * 2, 0x00020000);
    unsigned rr = lane >> 3, ch = lane & 7;
    unsigned ra = eb + rr * 128 + ((ch ^ rr) << 4);
    unsigned oo = (unsigned)(mb * 128 + 32 * cw + rr) * 8192u + nb * 256u + h * 128u + ch * 16u;
    #pragma unroll
    for (int i = 0; i < 4; ++i) {
      v4u o = lds_r128v(ra + i * 1024);
      __builtin_amdgcn_raw_buffer_store_b128(o, rO, oo, i * 8u * 8192u, 0);
    }
  }
}

extern "C" void kernel_launch(int rank, int nranks, ncclComm_t comm,
                              hipStream_t stream,
                              void* const* d_in, const int* in_sizes, int n_in,
                              void* d_out, int out_size,
                              const int* in_dtypes, int out_dtype,
                              void* const* d_out_all,
                              void* const* d_ws_all, size_t ws_bytes) {
  if (nranks != NR || ws_bytes < WS_NEED + WS_GUARD || (unsigned)rank >= (unsigned)NR) {
    fprintf(stderr, "unsupported configuration\n");
    return;
  }
  Args a;
  for (int p = 0; p < NR; ++p) a.ws[p] = (char*)d_ws_all[p];
  fused<<<256, 64 * NWAVE, 0, stream>>>(a, rank, (const char*)d_in[0], (const char*)d_in[1], (char*)d_out);
  HIP_CHECK(hipGetLastError());
}
```

```cpp
#include <hip/hip_runtime.h>
#include <rccl/rccl.h>
#include <cstdio>

#define HIP_CHECK(expr) do { hipError_t _e = (expr); if (_e != hipSuccess) { \
    fprintf(stderr, "HIP error %s at %s:%d\n", hipGetErrorString(_e), __FILE__, __LINE__); \
    return; } } while (0)

typedef unsigned v4u __attribute__((ext_vector_type(4)));
typedef unsigned v2u __attribute__((ext_vector_type(2)));
typedef float f32x16 __attribute__((ext_vector_type(16)));
typedef short s16x4 __attribute__((ext_vector_type(4)));
typedef short s16x8 __attribute__((ext_vector_type(8)));
typedef __bf16 bf16x8 __attribute__((ext_vector_type(8)));
typedef __amdgpu_buffer_rsrc_t rsrc_t;
typedef __attribute__((address_space(3))) s16x4* lds_s16x4_p;
typedef volatile __attribute__((address_space(3))) v4u* lds_vv4u_p;
typedef volatile __attribute__((address_space(3))) unsigned* lds_vu_p;

#define LD 8
#define TAU 350
#define PER 620
#define TBASE 150
#define NOPUSH 0
#define PLDK 8
#define PSLEEP 2
#define CPRIO 2
#define NLW 3
#define LPRIO 0
#define XP 0
#define AUX 17
#define NOA 0
#define RW 8
#define RFM 0x1222

constexpr int NR = 8;
constexpr int NWAVE = 8;
constexpr unsigned SLOT_BYTES = 4u << 20;
constexpr unsigned SRC_BYTES = 512u << 10;
constexpr unsigned PART_OFF = 3 * SLOT_BYTES;
constexpr unsigned FLAG_OFF = PART_OFF + 256 * 32768;
constexpr unsigned HINT_OFF = FLAG_OFF + 4096;
constexpr unsigned TMO_OFF = HINT_OFF + 4096;
constexpr unsigned CTR_OFF = TMO_OFF + 4096;
constexpr unsigned TS_OFF = CTR_OFF + 8192;
constexpr unsigned WS_NEED = TS_OFF + 256 * 16 * 16 * 8;
constexpr unsigned WS_GUARD = 2u << 20;
#define DL 100000000ull
constexpr unsigned long long DEADLINE_TICKS = DL;

constexpr int LDS_TOTAL = 160 * 1024;
constexpr int NU = 16 + 7 * 16;
constexpr unsigned CTRL = 39 * 4096;
constexpr unsigned C_FL = CTRL;
constexpr unsigned C_DONE = CTRL + 512;
constexpr unsigned C_BAD = CTRL + 1008;
constexpr unsigned C_ARR = CTRL + 1024;
constexpr int gslot(int r) { return r % 5 == 0 ? 32 : (r % 5 == 1 ? 0 : (r % 5 == 2 ? 16 : (r % 5 == 3 ? 7 : 23))); }
constexpr int hbit(int j) { return (j / 7) * 8 + j % 7; }
constexpr unsigned issued(int k, bool tail) {
  unsigned m = 0;
  for (int j = 0; j < 14; ++j) {
    if (j >= k && !(k == 0 && j == 13)) m |= 1u << hbit(j);
    if (!tail && j <= k - 2) m |= 1u << (16 + hbit(j));
  }
  return m;
}
constexpr unsigned notyet(int k) {
  unsigned m = k == 0 ? 1u << hbit(13) : 0u;
  for (int j = 0; j < 14; ++j) if (j > k - 2) m |= 1u << (16 + hbit(j));
  return m;
}

__device__ __forceinline__ unsigned phys(unsigned pos, unsigned mb) { return (pos & ~(RW - 1u)) | ((pos + mb * (RW / 4u)) & (RW - 1u)); }
struct Args { char* ws[NR]; };
template <int V> struct IC { static constexpr int value = V; };
template <int B, int E, typename F> __device__ __forceinline__ void sfor(F&& f) {
  if constexpr (B < E) { f(IC<B>{}); sfor<B + 1, E>(f); }
}

__device__ __forceinline__ unsigned pk_bf16(float a, float b) {
  unsigned r; asm("v_cvt_pk_bf16_f32 %0, %1, %2" : "=v"(r) : "v"(a), "v"(b)); return r;
}
__device__ __forceinline__ float silu(float y) {
  float e = __builtin_amdgcn_exp2f(y * -1.44269504088896341f);
  return y * __builtin_amdgcn_rcpf(1.f + e);
}
__device__ __forceinline__ unsigned rfl(unsigned v) { return __builtin_amdgcn_readfirstlane(v); }
__device__ __forceinline__ unsigned long long now() { return __builtin_amdgcn_s_memrealtime(); }
__device__ __forceinline__ void lds_w32(unsigned addr, unsigned v) { *(lds_vu_p)(size_t)addr = v; }
__device__ __forceinline__ unsigned lds_r32(unsigned addr) { return *(lds_vu_p)(size_t)addr; }
__device__ __forceinline__ v4u lds_r128v(unsigned addr) { return *(lds_vv4u_p)(size_t)addr; }
__device__ __forceinline__ void glds4(rsrc_t r, unsigned v0, unsigned v1, unsigned s0, unsigned s1, unsigned s2, unsigned s3, unsigned la) {
  asm volatile("s_mov_b32 m0, %0\n\ts_nop 0\n\t"
               "buffer_load_dwordx4 %1, %3, %4 offen lds\n\t"
               "buffer_load_dwordx4 %2, %3, %5 offen offset:1024 lds\n\t"
               "buffer_load_dwordx4 %1, %3, %6 offen offset:2048 lds\n\t"
               "buffer_load_dwordx4 %2, %3, %7 offen offset:3072 lds"
               :: "s"(la), "v"(v0), "v"(v1), "s"(r), "s"(s0), "s"(s1), "s"(s2), "s"(s3) : "m0", "memory");
}
__device__ __forceinline__ void gzero(rsrc_t r, unsigned zo, unsigned la) {
  asm volatile("s_mov_b32 m0, %0\n\ts_mov_b64 exec, 1\n\tbuffer_load_dword off, %1, %2 lds\n\ts_mov_b64 exec, -1"
               :: "s"(la), "s"(r), "s"(zo) : "m0", "memory");
}
__device__ __forceinline__ void st1(rsrc_t r, unsigned so, unsigned v) {
  asm volatile("s_mov_b64 exec, 1\n\tbuffer_store_dword %0, off, %1, %2 sc0 sc1\n\ts_mov_b64 exec, -1" :: "v"(v), "s"(r), "s"(so) : "memory");
}
__device__ __forceinline__ void reld(v4u& A, rsrc_t r, unsigned vo, unsigned so) {
  const unsigned long long z = __builtin_amdgcn_ballot_w64(min(min(A[0], A[1]), min(A[2], A[3])) == 0);
  asm volatile("s_mov_b64 exec, %1\n\tbuffer_load_dwordx4 %0, %2, %3, %4 offen sc0 sc1\n\ts_mov_b64 exec, -1"
               : "+v"(A) : "s"(z), "v"(vo), "s"(r), "s"(so) : "memory");
}
__device__ __forceinline__ void pin(v4u& A) { asm volatile("" : "+v"(A)); }
__device__ __forceinline__ void sor(unsigned& A, unsigned b) { asm volatile("s_or_b32 %0, %0, %1" : "+s"(A) : "s"(b) : "scc"); }
#define VMCNT(n) asm volatile("s_waitcnt vmcnt(%0)" :: "n"(n) : "memory")
#define SB() __builtin_amdgcn_sched_barrier(0)
#define MEMB() asm volatile("" ::: "memory")
#define BARRIER() asm volatile("s_barrier" ::: "memory")
__device__ __forceinline__ unsigned min4(const v4u& a) { return min(min(a[0], a[1]), min(a[2], a[3])); }
__device__ __forceinline__ bool anyz(unsigned v) { return __builtin_amdgcn_ballot_w64(v == 0) != 0; }

#define STAMP(k) do {} while (0)
#define TSOUT() do {} while (0)

__global__ __launch_bounds__(64 * NWAVE, 1) void fused(Args a, int rank, const char* __restrict__ x,
                                                       const char* __restrict__ W, char* __restrict__ out) {
  __shared__ __attribute__((aligned(1024))) char smem[LDS_TOTAL];
  const int tid = threadIdx.x, lane = tid & 63, w = rfl(tid >> 6);
  const int bid = blockIdx.x;
  const unsigned long long T0 = now();
  const int g = bid & 7, idx = bid >> 3;
  const int h = g & 1, nb = (idx & 7) * 4 + (g >> 1), mb = idx >> 3;
  const int rot = (idx + (idx >> 3)) & 3;
  const int hh = lane >> 5, m = lane & 31;
  char* me = a.ws[rank];
  const rsrc_t rX = __builtin_amdgcn_make_buffer_rsrc((void*)x, (short)0, 4096 * 512 * 2, 0x00020000);
  const rsrc_t rMe = __builtin_amdgcn_make_buffer_rsrc((void*)me, (short)0, (int)WS_NEED, 0x00020000);
  const unsigned lbase = (unsigned)(size_t)(__attribute__((address_space(3))) char*)smem;
  const unsigned co = CTR_OFF + (bid * 8 + w) * 4;
  auto timeout = [&](unsigned call) { st1(rMe, TMO_OFF, call); lds_w32(lbase + C_BAD, 0); };

  if (w == 0 && NLW < 4) {
    const int p = (rank + 1 + bid % 7) & 7, q = bid / 7;
    if (bid < 224 && !NOPUSH) {
      v4u v[16];
      const unsigned xv = m * 1024u + hh * 16u;
      const unsigned xs = (unsigned)(p * 512 + (q >> 1) * 32) * 1024u + (q & 1) * 32u;
      const unsigned cv = __builtin_amdgcn_raw_buffer_load_b32(rMe, 0, co, 17);
      #pragma unroll
      for (int j = 0; j < 16; ++j) v[j] = __builtin_amdgcn_raw_buffer_load_b128(rX, xv, xs + phys(j, q >> 3) * 64, 0);
      BARRIER();
      const unsigned call = rfl(cv) + 1;
      const rsrc_t rP = __builtin_amdgcn_make_buffer_rsrc((void*)a.ws[p], (short)0, (int)WS_NEED, 0x00020000);
      const unsigned so = (call % 3) * SLOT_BYTES + rank * SRC_BYTES + (q & 1) * 16384 + (q >> 1) * 1024;
      const unsigned vo = lane * 16;
      const unsigned long long t1 = T0 + TBASE + (unsigned)(q * PER) / 320;
      STAMP(1);
      #pragma unroll
      for (int j = 0; j < 16; ++j) {
        const unsigned long long tq = t1 + (j * PER) / 10;
        while ((long long)(now() - tq) < 0) __builtin_amdgcn_s_sleep(1);
        if (j == 0) STAMP(2);
        asm volatile("buffer_store_dwordx4 %0, %1, %2, %3 offen sc0 sc1"
                     :: "v"(v[j]), "v"(vo), "s"(rP), "s"(so + j * 32768) : "memory");
      }
      asm volatile("s_nop 1" :: "v"(v[15]));
      st1(rMe, co, call);
      STAMP(4);
      VMCNT(0);
      STAMP(3);
      st1(rP, FLAG_OFF + (rank * 32 + q) * 4, call);
    } else {
      BARRIER();
    }
    TSOUT();
    return;
  }
  if (w == 3) {
      const unsigned za = lbase + CTRL + lane * 16;
      const v4u ones = {~0u, ~0u, ~0u, ~0u};
      asm volatile("ds_write_b128 %0, %1\n\tds_write_b128 %0, %1 offset:1024\n\ts_waitcnt lgkmcnt(0)" :: "v"(za), "v"(ones) : "memory");
  }
  if (w == 1 || w == 2 || (NLW > 2 && w == 3) || (NLW > 3 && w == 0)) {
    const int lw = (w + 3) & 3;
    if (LPRIO) __builtin_amdgcn_s_setprio(LPRIO);
    const rsrc_t rW = __builtin_amdgcn_make_buffer_rsrc((void*)W, (short)0, 4096 * 4096 * 2, 0x00020000);
    const unsigned wq = lane >> 4;
    const unsigned ch = (lane & 15) ^ (((wq >> 1) << 3) | ((wq & 1) << 1));
    const unsigned wv1 = wq * 8192u + ((ch ^ (8u * h)) << 4) + nb * 256u;
    const unsigned wv2 = (wq + ((ch >> 3) ^ h) * 16) * 8192u + ((ch & 7) << 4) + nb * 256u + h * 128u;
    const unsigned cv = __builtin_amdgcn_raw_buffer_load_b32(rMe, 0, co, 17);
    int seen = 0;
    int rr = 0, uu = lw - 16, g5 = 0;
    BARRIER();
    if (XP & 1) return;
    for (int n = lw; n < NU; n += NLW) {
      unsigned k0, wv = wv1, slot;
      int need = 0;
      if (n < 16) { k0 = rank * 512 + (((n >> 2) + rot) & 3) * 128 + 64 * h + 16 * (n & 3); slot = n; }
      else {
        k0 = ((rank + 1 + uu) & 7) * 512 + 32 * phys(rr, mb);
        if (rr < 14) k0 += 16 * h; else wv = wv2;
        slot = ((0x1707100020ull >> (8 * g5)) & 255) + uu;
        need = rr < 5 ? (rr + 1) >> 1 : rr - 2;
      }
      if (seen < need) {
        int spin = 0;
        while (true) {
          const v4u d = lds_r128v(lbase + C_DONE + (need - 1) * 16);
          if (rfl(d[0] | d[1] | d[2] | d[3]) == 0) break;
          if ((++spin & 255) == 0 && now() - T0 > 2 * DEADLINE_TICKS) break;
          __builtin_amdgcn_s_sleep(1);
        }
        seen = need;
      }
      const unsigned so = k0 * 8192u;
      glds4(rW, wv, wv, so, so + 31744u, so + 2 * 31744u, so + 3 * 31744u, lbase + slot * 4096);
      gzero(rMe, WS_NEED, lbase + C_FL + n * 4);
      VMCNT(5 * (LD - 1));
      uu += NLW;
      if (uu >= 7) { uu -= 7; ++rr; g5 = g5 == 4 ? 0 : g5 + 1; }
    }
    if (lw == 0 && !NOPUSH) {
      const unsigned call = rfl(cv) + 1;
      const unsigned zo = ((call + 2) % 3) * SLOT_BYTES + bid * 16384, vo = lane * 16;
      const v4u zero = {0, 0, 0, 0};
      #pragma unroll
      for (int j = 0; j < 16; ++j)
        asm volatile("buffer_store_dwordx4 %0, %1, %2, %3 offen offset:%4 sc0 sc1"
                     :: "v"(zero), "v"(vo), "s"(rMe), "s"(zo + (j >> 2) * 4096), "n"((j & 3) * 1024) : "memory");
      asm volatile("s_nop 1" :: "v"(zero));
      st1(rMe, co, call);
    }
    VMCNT(0);
    return;
  }
  const int cw = w - 4;
  const unsigned cv = __builtin_amdgcn_raw_buffer_load_b32(rMe, 0, co, 17);
  BARRIER();
  if (XP & 1) { TSOUT(); return; }
  {
    const unsigned xv = (lane >> 3) * 1024u + (((lane & 7) ^ (lane >> 4)) << 4);
    const unsigned xs = (unsigned)(rank * 512 + mb * 128 + 32 * cw) * 1024u + 128u * h;
    #pragma unroll
    for (int b = 0; b < 4; ++b) {
      const unsigned so = xs + ((b + rot) & 3) * 256u;
      glds4(rX, xv, xv ^ 64u, so, so + 7168u, so + 2 * 7168u, so + 3 * 7168u, lbase + (16 + 4 * b + cw) * 4096);
    }
  }
  if (CPRIO) __builtin_amdgcn_s_setprio(CPRIO);

  f32x16 acc[4];
  #pragma unroll
  for (int i = 0; i < 4; ++i)
    #pragma unroll
    for (int j = 0; j < 16; ++j) acc[i][j] = 0.f;

  unsigned tb[4], tw[4];
  {
    const unsigned q = (lane >> 2) & 3, p = lane & 3, blk = (lane >> 4) & 1;
    unsigned b0 = 256u * (8 * hh + q) + 128u * (q >> 1) + 32u * (q & 1) + 64u * (p & 1) + 16u * blk + 8u * (p >> 1);
    tb[0] = lbase + b0; tb[1] = lbase + (b0 ^ 32u); tb[2] = lbase + (b0 ^ 128u); tb[3] = lbase + (b0 ^ 160u);
  }
  const unsigned xra = lbase + (16 + cw) * 4096 + m * 128 + (((unsigned)hh ^ ((m >> 1) & 7)) << 4);
  const unsigned avo = (unsigned)(mb * 4 + cw) * 1024u + lane * 16 + h * 16384;
  unsigned fad = lbase + C_FL;
  unsigned vn0 = avo, vn1 = avo + 32768;
  asm volatile("" : "+v"(fad));
  unsigned dad = lbase + C_DONE + 4 * cw;
  s16x4 wf[2][8];
  v4u B[14], Y[14];
  unsigned F = 0;
  VMCNT(16);
  const unsigned call = rfl(cv) + 1;
  st1(rMe, co, call);
  unsigned su[7];
  #pragma unroll
  for (int u = 0; u < 7; ++u) su[u] = (call % 3) * SLOT_BYTES + ((rank + 1 + u) & 7) * SRC_BYTES;

  auto rdtr = [&](unsigned ad) { return __builtin_amdgcn_ds_read_tr16_b64_v4i16((lds_s16x4_p)(size_t)ad); };
  auto mma = [&](const s16x4 (&f)[8], int i, int ai, const v4u& b) {
    s16x8 aa = __builtin_shufflevector(f[2 * i], f[2 * i + 1], 0, 1, 2, 3, 4, 5, 6, 7);
    acc[ai] = __builtin_amdgcn_mfma_f32_32x32x16_bf16(__builtin_bit_cast(bf16x8, aa), __builtin_bit_cast(bf16x8, b), acc[ai], 0, 0, 0);
  };
  auto ldA = [&](unsigned vo, int u, int bit) -> v4u {
    if (NOA == 2) { v4u z; asm volatile("" : "=v"(z)); return z; }
    const unsigned so = NOA ? WS_NEED : su[u];
    return __builtin_amdgcn_raw_buffer_load_b128(rMe, vo, so, AUX);
  };
  auto isbad = [&]() { return rfl(lds_r32(lbase + C_BAD)) == 0; };
  auto gslow = [&](v4u& A, unsigned vo, int u, int hs) {
    const unsigned fo = lane == 0 ? FLAG_OFF + (((rank + 1 + u) & 7) * 32 + 2 * (mb * 4 + cw) + hs) * 4 : TMO_OFF;
    bool fseen = false;
    #pragma nounroll
    while (true) {
      unsigned fv;
      asm volatile("s_nop 4\n\tbuffer_load_dwordx4 %0, %2, %3, %4 offen sc0 sc1\n\t"
                   "buffer_load_dword %1, %5, %3, 0 offen sc0 sc1\n\ts_waitcnt vmcnt(0)"
                   : "+&v"(A), "=&v"(fv) : "v"(vo), "s"(rMe), "s"(su[u]), "v"(fo) : "memory");
      if (fseen || !anyz(min4(A))) break;
      fseen = (__builtin_amdgcn_ballot_w64((int)(fv - call) >= 0) & 1) != 0;
      const bool tmo_hit = (__builtin_amdgcn_ballot_w64(fv == call) >> 1) & 1;
      if (tmo_hit || now() - T0 > DEADLINE_TICKS) { timeout(call); break; }
      __builtin_amdgcn_s_sleep(PSLEEP);
    }
  };
  auto fspin = [&](unsigned fa) {
    int spin = 0;
    #pragma nounroll
    while (rfl(lds_r32(fa)) != 0) {
      if ((++spin & 31) == 0 && now() - T0 > 2 * DEADLINE_TICKS) { lds_w32(lbase + C_BAD, 0); break; }
    }
  };
  auto step = [&](auto MODE_, auto K_, int rb) {
    constexpr int MODE = decltype(MODE_)::value, K = decltype(K_)::value;
    constexpr int PB = (K + 1) & 1;
    constexpr bool LAST = MODE == 3 && K == 14;
    constexpr bool CT = MODE == 3 && K > 0;
    constexpr bool VAL = MODE != 0 && !LAST && !NOPUSH;
    constexpr bool FCK = !(MODE == 3 && K >= 13);
    constexpr int CB = MODE == 0 ? 12 + ((K + 1) & 1) : (K + 13) % 14;
    constexpr unsigned woff = MODE == 0 ? K * 4096 : (K % 7) * 4096;
    s16x4 (&wc)[8] = wf[PB];
    s16x4 (&wn)[8] = wf[PB ^ 1];
    unsigned t = 1;
    bool fail = false;
    auto tr = [&](int j) { if constexpr (!LAST) wn[j] = rdtr(tw[j >> 1] + woff + (j & 1) * 1024); };
    #pragma unroll
    for (int i = 0; i < 4; ++i) {
      SB();
      if (i == 0) {
        if constexpr (MODE == 0) B[12 + (K & 1)] = lds_r128v((xra ^ ((K & 3) << 5)) + (K >> 2) * 16384);
        tr(0); tr(1);
        if constexpr (MODE == 0) { if constexpr (K >= 4) B[K - 4] = ldA(K - 4 < 7 ? vn0 : vn1, (K - 4) % 7, hbit(K - 4)); }
        else if constexpr (K < 2) {
          B[12 + K] = ldA(vn1, 5 + K, hbit(12 + K));
          if constexpr (MODE == 3) Y[12 + K] = ldA(vn1 ^ 16384u, 5 + K, hbit(12 + K));
        } else if constexpr (MODE != 3) {
          B[K - 2] = ldA(K - 2 < 7 ? vn0 : vn1, (K - 2) % 7, 16 + hbit(K - 2));
          if constexpr (MODE == 2) Y[K - 2] = ldA((K - 2 < 7 ? vn0 : vn1) ^ 16384u, (K - 2) % 7, 16 + hbit(K - 2));
        }
      }
      if (i == 1) {
        tr(2); tr(3); tr(4);
        if constexpr (VAL) {
          t = min4(B[K]);
          if constexpr (MODE == 3) t = min(min(min(t, Y[K][0]), Y[K][1]), min(Y[K][2], Y[K][3]));
        }
        if constexpr (MODE == 1 || MODE == 2) { if (K == 0) vn0 += 65536; if (K == 2) vn1 += 65536; }
      }
      if (i == 2) {
        tr(5); tr(6); tr(7);
        if constexpr (FCK) fail = __builtin_amdgcn_ballot_w64(t <= F) != 0;
        else if constexpr (VAL) fail = anyz(t);
      }
      if (i == 3) {
        if (__builtin_expect(fail, 0)) {
          if constexpr (FCK) fspin(fad + (K + 1) * 4);
          if constexpr (VAL) {
            if (anyz(t) && !isbad()) {
              const unsigned v0 = avo + rb * 32768;
              auto all = [&]() {
                VMCNT(0);
                sfor<0, 14>([&](auto JC) {
                  constexpr int j = (decltype(JC)::value + K) % 14;
                  if constexpr (j >= K && !(K == 0 && j == 13)) {
                    reld(B[j], rMe, v0, su[j % 7] + (j / 7) * 32768);
                    if constexpr (MODE == 3) reld(Y[j], rMe, v0 ^ 16384u, su[j % 7] + (j / 7) * 32768);
                  }
                  if constexpr (MODE != 3 && j <= K - 2) {
                    reld(B[j], rMe, v0, su[j % 7] + (2 + j / 7) * 32768);
                    if constexpr (MODE == 2) reld(Y[j], rMe, v0 ^ 16384u, su[j % 7] + (2 + j / 7) * 32768);
                  }
                });
                VMCNT(0);
              };
              #pragma nounroll
              for (int pass = 0; pass < 2; ++pass) {
                all();
                pin(B[K]);
                if (!anyz(min4(B[K]))) break;
                if (pass == 0) gslow(B[K], v0 + (K / 7) * 32768, K % 7, h);
              }
              if constexpr (MODE == 3) {
                pin(Y[K]);
                if (anyz(min4(Y[K]))) gslow(Y[K], (v0 + (K / 7) * 32768) ^ 16384u, K % 7, h ^ 1);
              }
            }
          }
        }
        SB();
        if constexpr (!(MODE == 3 && K >= 12)) F = lds_r32(fad + (K + 2) * 4);
        if constexpr (MODE == 0 ? K == 9 : (MODE == 1 && K % 7 == 1)) {
          gzero(rMe, WS_NEED, dad); dad += 16;
        }
      }
      SB();
      if constexpr (CT) mma(wc, i, i & 1, i < 2 ? B[CB] : Y[CB]);
      else mma(wc, i, i, B[CB]);
    }
    SB();
  };

  VMCNT(13);
  B[12] = lds_r128v(xra);
  fspin(fad);
  fspin(fad + 4);
  STAMP(1);
  #pragma unroll
  for (int i = 0; i < 4; ++i) { tw[i] = tb[i]; wf[0][2 * i] = rdtr(tw[i]); wf[0][2 * i + 1] = rdtr(tw[i] + 1024); }
  F = lds_r32(fad + 8);
  VMCNT(1);
  sfor<1, 16>([&](auto NC) { step(IC<0>{}, NC, 0); });
  STAMP(2);
  fad += 16 * 4;
  auto iter = [&](auto MODE_, int r) {
    sfor<0, 14>([&](auto KC) {
      constexpr int k = decltype(KC)::value;
      if constexpr (k % 7 == 0) {
        const unsigned go = ((unsigned)(0x201707100020ull >> (8 * ((r % 5) + k / 7))) & 255) * 4096;
        #pragma unroll
        for (int i = 0; i < 4; ++i) tw[i] = tb[i] + go;
      }
      step(MODE_, KC, r);
    });
    fad += 14 * 4;
  };
  for (int r = 0; r < 12; r += 2) {
    iter(IC<1>{}, r);
  }
  STAMP(6);
  iter(IC<2>{}, 12);
  STAMP(7);
  v4u P[8];
  const unsigned pvo = lane * 16;
  sfor<0, 15>([&](auto KC) {
    constexpr int k = decltype(KC)::value;
    if constexpr (k % 7 == 0 && k < 14) {
      #pragma unroll
      for (int i = 0; i < 4; ++i) tw[i] = tb[i] + gslot(14 + k / 7) * 4096;
    }
    step(IC<3>{}, KC, 14);
    if constexpr (k == 0) {
      const unsigned po = PART_OFF + bid * 32768 + cw * 8192;
      const bool bad = isbad();
      const unsigned am = bad ? 0u : ~1u, ov = (bad ? 0x7FC00000u : 0u) | (call & 1);
      v4u t[8];
      #pragma unroll
      for (int c = 0; c < 8; ++c)
        #pragma unroll
        for (int e = 0; e < 4; ++e) { const float f = acc[2 + (c >> 2)][4 * (c & 3) + e]; t[c][e] = (__float_as_uint(f) & am) | ov; }
      asm volatile("buffer_store_dwordx4 %0, %8, %9, %10 offen sc0 sc1\n\t"
                   "buffer_store_dwordx4 %1, %8, %9, %10 offen offset:1024 sc0 sc1\n\t"
                   "buffer_store_dwordx4 %2, %8, %9, %10 offen offset:2048 sc0 sc1\n\t"
                   "buffer_store_dwordx4 %3, %8, %9, %10 offen offset:3072 sc0 sc1\n\t"
                   "buffer_store_dwordx4 %4, %8, %9, %11 offen sc0 sc1\n\t"
                   "buffer_store_dwordx4 %5, %8, %9, %11 offen offset:1024 sc0 sc1\n\t"
                   "buffer_store_dwordx4 %6, %8, %9, %11 offen offset:2048 sc0 sc1\n\t"
                   "buffer_store_dwordx4 %7, %8, %9, %11 offen offset:3072 sc0 sc1\n\ts_nop 1"
                   :: "v"(t[0]), "v"(t[1]), "v"(t[2]), "v"(t[3]), "v"(t[4]), "v"(t[5]), "v"(t[6]), "v"(t[7]),
                      "v"(pvo), "s"(rMe), "s"(po), "s"(po + 4096) : "memory");
    }
    if constexpr (k == PLDK) {
      const unsigned po = PART_OFF + (bid ^ 1) * 32768 + cw * 8192;
      #pragma unroll
      for (int c = 0; c < 8; ++c) P[c] = __builtin_amdgcn_raw_buffer_load_b128(rMe, pvo + (c & 3) * 1024, po + (c >> 2) * 4096, 17);
    }
  });
  STAMP(9);

  bool bad = isbad();
  {
    const unsigned po = PART_OFF + (bid ^ 1) * 32768 + cw * 8192;
    const unsigned tag = call & 1;
    int it = 0;
    #pragma nounroll
    while (true) {
      unsigned o0 = 0;
      #pragma unroll
      for (int c = 0; c < 8; ++c)
        #pragma unroll
        for (int e = 0; e < 4; ++e) o0 |= P[c][e] ^ tag;
      if (__builtin_amdgcn_ballot_w64((o0 & 1) != 0) == 0 || bad) break;
      if ((++it & 15) == 0) {
        const unsigned tv = rfl(__builtin_amdgcn_raw_buffer_load_b32(rMe, 0, TMO_OFF, 17));
        if (tv == call || now() - T0 > DEADLINE_TICKS) { timeout(call); bad = true; }
      }
      MEMB();
      #pragma unroll
      for (int c = 0; c < 8; ++c) P[c] = __builtin_amdgcn_raw_buffer_load_b128(rMe, pvo + (c & 3) * 1024, po + (c >> 2) * 4096, 17);
    }
  }
  STAMP(10);
  const unsigned nanm = bad ? 0x7FC07FC0u : 0u;
  {
    const unsigned eb = lbase + (14 + (cw & 1) + (cw >> 1) * 16) * 4096;
    const unsigned ta = eb + m * 128;
    #pragma unroll
    for (int i = 0; i < 2; ++i)
      #pragma unroll
      for (int c2 = 0; c2 < 2; ++c2) {
        v4u t;
        #pragma unroll
        for (int e = 0; e < 4; ++e) {
          const int j = 8 * c2 + 2 * e;
          const unsigned p0 = P[i * 4 + (j >> 2)][j & 3], p1 = P[i * 4 + (j >> 2)][(j & 3) + 1];
          float y0 = acc[i][j] + __uint_as_float(p0), y1 = acc[i][j + 1] + __uint_as_float(p1);
          t[e] = pk_bf16(silu(y0), silu(y1)) | nanm;
        }
        *(lds_vv4u_p)(size_t)(ta + (((4 * hh + 2 * i + c2) ^ (m & 7)) << 4)) = t;
      }
    const rsrc_t rO = __builtin_amdgcn_make_buffer_rsrc((void*)out, (short)0, 512 * 4096 * 2, 0x00020000);
    unsigned rr = lane >> 3, ch = lane & 7;
    unsigned ra = eb + rr * 128 + ((ch ^ rr) << 4);
    unsigned oo = (unsigned)(mb * 128 + 32 * cw + rr) * 8192u + nb * 256u + h * 128u + ch * 16u;
    #pragma unroll
    for (int i = 0; i < 4; ++i) {
      v4u o = lds_r128v(ra + i * 1024);
      __builtin_amdgcn_raw_buffer_store_b128(o, rO, oo, i * 8u * 8192u, 0);
    }
  }
}

extern "C" void kernel_launch(int rank, int nranks, ncclComm_t comm,
                              hipStream_t stream,
                              void* const* d_in, const int* in_sizes, int n_in,
                              void* d_out, int out_size,
                              const int* in_dtypes, int out_dtype,
                              void* const* d_out_all,
                              void* const* d_ws_all, size_t ws_bytes) {
  if (nranks != NR || ws_bytes < WS_NEED + WS_GUARD || (unsigned)rank >= (unsigned)NR) {
    fprintf(stderr, "unsupported configuration\n");
    return;
  }
  Args a;
  for (int p = 0; p < NR; ++p) a.ws[p] = (char*)d_ws_all[p];
  fused<<<256, 64 * NWAVE, 0, stream>>>(a, rank, (const char*)d_in[0], (const char*)d_in[1], (char*)d_out);
  HIP_CHECK(hipGetLastError());
}
```

```cpp
#include <hip/hip_runtime.h>
#include <rccl/rccl.h>
#include <cstdio>

#define HIP_CHECK(expr) do { hipError_t _e = (expr); if (_e != hipSuccess) { \
    fprintf(stderr, "HIP error %s at %s:%d\n", hipGetErrorString(_e), __FILE__, __LINE__); \
    return; } } while (0)

typedef unsigned v4u __attribute__((ext_vector_type(4)));
typedef unsigned v2u __attribute__((ext_vector_type(2)));
typedef float f32x16 __attribute__((ext_vector_type(16)));
typedef float f32x2 __attribute__((ext_vector_type(2)));
typedef short s16x4 __attribute__((ext_vector_type(4)));
typedef short s16x8 __attribute__((ext_vector_type(8)));
typedef __bf16 bf16x8 __attribute__((ext_vector_type(8)));
typedef __amdgpu_buffer_rsrc_t rsrc_t;
typedef unsigned long long u64;
typedef __attribute__((address_space(3))) s16x4* lds_s16x4_p;
typedef volatile __attribute__((address_space(3))) v4u* lds_vv4u_p;
typedef volatile __attribute__((address_space(3))) v2u* lds_vv2u_p;
typedef volatile __attribute__((address_space(3))) unsigned* lds_vu_p;

#define LD 12
#define PER 700
#define TBASE 110
#define NOPUSH 0
#define PLDK 7
#define PSLEEP 2
#define CPRIO 2
#define LPRIO 0
#define SPRIO 0
#define SPRIO2 0
#define PPRIO 0
#define RW 4
#define OAUX 16
#define PF0 4
#define TLD 130
#define PP 25
#define DA 140
#define SL2 20
#define LEAD 30
#define ESL 2
#define SSL 2
#define SSL2 8
#define SSD 2
#define NFINE 21
#define PW1 30
#define ES0 0
#define WARM 2
#define TOUCH 0
#define LSL 0
#define WPRIO 2
#define STR_(x) #x
#define STR(x) STR_(x)
#define PM0 0xEE
#define PM1 0x55
#define PM2 0x11
#define PM4 0xB6
#define TOPO 0x4D
constexpr unsigned long long PMASK = PM0 | PM1 << 8 | PM2 << 16 | (unsigned long long)(255 ^ PM0 ^ PM2) << 24 | (PM4 | (255ull ^ PM4) << 8) * 0x100010000ull << 16;
#define XS 0
#define PX 0

constexpr int NR = 8;
constexpr int NWAVE = 8;
constexpr unsigned SLOT_BYTES = 4u << 20;
constexpr unsigned SRC_BYTES = 512u << 10;
constexpr unsigned PART_OFF = 0;
constexpr unsigned SLOT_OFF = 8u << 20;
#define FILL_Q0 2
constexpr unsigned FLAG_OFF = SLOT_OFF + 3 * SLOT_BYTES;
constexpr unsigned HINT_OFF = FLAG_OFF + 4096;
constexpr unsigned TMO_OFF = HINT_OFF + 32768;
constexpr unsigned DIRTY_OFF = TMO_OFF + 1024;
constexpr unsigned CTR_OFF = TMO_OFF + 4096;
constexpr unsigned TS_OFF = CTR_OFF + 8192;
constexpr unsigned WS_NEED = TS_OFF + 256 * 16 * 16 * 8;
constexpr unsigned WS_GUARD = 2u << 20;
#define DL 100000000ull
constexpr unsigned long long DEADLINE_TICKS = DL;

constexpr int LDS_TOTAL = 160 * 1024;
constexpr unsigned CTRL = 39 * 4096;
constexpr unsigned C_FL = CTRL;
constexpr unsigned C_AL = C_FL + 36;
constexpr unsigned C_WL = CTRL + 2048;
constexpr unsigned C_DONE = CTRL + 2560;
constexpr unsigned C_BAD = CTRL + 2880;
constexpr unsigned C_DRT = C_BAD + 4;
constexpr unsigned C_XF = CTRL + 2944;
constexpr unsigned C_ARR = CTRL + 3072;
constexpr unsigned NONE = ~0u;
constexpr int NEVER = 0x40000000;
constexpr int gslot(int r) { return r % 5 == 0 ? 32 : (r % 5 == 1 ? 0 : (r % 5 == 2 ? 16 : (r % 5 == 3 ? 7 : 23))); }
constexpr int BT(int b) { return b == 0 ? 0 : b == 1 ? 4 : b == 2 ? 7 : b == 3 ? 10 : 12; }

__device__ __forceinline__ unsigned phys(unsigned pos, unsigned mb) { return pos ^ (mb * (RW / 4u)); }
struct Args { char* ws[NR]; };
template <int V> struct IC { static constexpr int value = V; };
template <int B, int E, typename F> __device__ __forceinline__ void sfor(F&& f) {
  if constexpr (B < E) { f(IC<B>{}); sfor<B + 1, E>(f); }
}

__device__ __forceinline__ unsigned pk_bf16(float a, float b) {
  unsigned r; asm("v_cvt_pk_bf16_f32 %0, %1, %2" : "=v"(r) : "v"(a), "v"(b)); return r;
}
__device__ __forceinline__ float silu(float y) {
  float e = __builtin_amdgcn_exp2f(y * -1.44269504088896341f);
  return y * __builtin_amdgcn_rcpf(1.f + e);
}
__device__ __forceinline__ unsigned rfl(unsigned v) { return __builtin_amdgcn_readfirstlane(v); }
__device__ __forceinline__ unsigned rdl(unsigned v, int l) { return __builtin_amdgcn_readlane(v, l); }
__device__ __forceinline__ u64 bal(bool b) { return __builtin_amdgcn_ballot_w64(b); }
__device__ __forceinline__ unsigned long long now() { return __builtin_amdgcn_s_memrealtime(); }
__device__ __forceinline__ void lds_w32(unsigned addr, unsigned v) { *(lds_vu_p)(size_t)addr = v; }
__device__ __forceinline__ unsigned lds_r32(unsigned addr) { return *(lds_vu_p)(size_t)addr; }
__device__ __forceinline__ v2u lds_r64(unsigned addr) { return *(lds_vv2u_p)(size_t)addr; }
__device__ __forceinline__ v4u lds_r128v(unsigned addr) { return *(lds_vv4u_p)(size_t)addr; }
#define TWP 1
#define QP 97
#define SPS 2
#define SPP 1
#define PLDK2 10
#define PNW 0
#define EOF_ 1
#define SPL 8
#define SX 0
#define PBI 0
#define PAUX 17
#define PST "sc0 sc1"
#define WPOL 0
#define XPOL 0
#define GLDS4(P) asm volatile("s_mov_b32 m0, %0\n\ts_nop 3\n\t" \
               "buffer_load_dwordx4 %1, %5, %6 offen " P " lds\n\t" \
               "buffer_load_dwordx4 %2, %5, %6 offen offset:1024 " P " lds\n\t" \
               "buffer_load_dwordx4 %3, %5, %6 offen offset:2048 " P " lds\n\t" \
               "buffer_load_dwordx4 %4, %5, %6 offen offset:3072 " P " lds" \
               :: "s"(la), "v"(v[0]), "v"(v[1]), "v"(v[2]), "v"(v[3]), "s"(r), "s"(so) : "m0", "memory")
template <int P> __device__ __forceinline__ void glds4p(rsrc_t r, const v4u& v, unsigned so, unsigned la) {
  if constexpr (P == 0) GLDS4(""); else if constexpr (P == 1) GLDS4("nt"); else if constexpr (P == 2) GLDS4("sc1");
  else if constexpr (P == 3) GLDS4("sc0 sc1"); else if constexpr (P == 4) GLDS4("sc0"); else if constexpr (P == 5) GLDS4("sc0 nt"); else GLDS4("sc1 nt");
}
__device__ __forceinline__ void gzero(rsrc_t r, unsigned zo, unsigned la) {
  asm volatile("s_mov_b32 m0, %0\n\ts_mov_b64 exec, 1\n\tbuffer_load_dword off, %1, %2 lds\n\ts_mov_b64 exec, -1"
               :: "s"(la), "s"(r), "s"(zo) : "m0", "memory");
}
__device__ __forceinline__ void st1(rsrc_t r, unsigned so, unsigned v) {
  asm volatile("s_mov_b64 exec, 1\n\tbuffer_store_dword %0, off, %1, %2 sc0 sc1\n\ts_mov_b64 exec, -1" :: "v"(v), "s"(r), "s"(so) : "memory");
}
__device__ __forceinline__ void reld(v4u& A, rsrc_t r, unsigned vo, unsigned so) {
  asm volatile("s_nop 4\n\tbuffer_load_dwordx4 %0, %1, %2, %3 offen sc0 sc1" : "+v"(A) : "v"(vo), "s"(r), "s"(so) : "memory");
}
__device__ __forceinline__ void pin(v4u& A) { asm volatile("" : "+v"(A)); }
#define PREGS "v208", "v209", "v210", "v211", "v212", "v213", "v214", "v215", "v216", "v217", "v218", "v219", "v220", "v221", "v222", "v223", "v224", "v225", "v226", "v227", "v228", "v229", "v230", "v231", "v232", "v233", "v234", "v235", "v236", "v237", "v238", "v239", "v240", "v241", "v242", "v243", "v244", "v245", "v246", "v247", "v248", "v249", "v250", "v251", "v252", "v253", "v254", "v255"
#define VMCNT(n) asm volatile("s_waitcnt vmcnt(%0)" :: "n"(n) : "memory")
#define SB() __builtin_amdgcn_sched_barrier(0)
#define MEMB() asm volatile("" ::: "memory")
#define BARRIER() asm volatile("s_barrier" ::: "memory")
__device__ __forceinline__ unsigned min4(const v4u& a) { return min(min(a[0], a[1]), min(a[2], a[3])); }
__device__ __forceinline__ unsigned min3(unsigned a, unsigned b, unsigned c) { unsigned r; asm("v_min3_u32 %0, %1, %2, %3" : "=v"(r) : "v"(a), "v"(b), "v"(c)); return r; }
__device__ __forceinline__ unsigned tagm(unsigned call) { return (((call - 1) / 3) & 1) << 29; }
__device__ __forceinline__ unsigned tagv(const v4u& a, unsigned k) { unsigned r; asm("v_lshl_add_u32 %0, %1, 1, %2" : "=v"(r) : "v"(a[0]), "s"(k)); return r; }
__device__ __forceinline__ bool anyp(unsigned v) { return __builtin_amdgcn_ballot_w64((int)v >= 0) != 0; }
__device__ __forceinline__ bool anyz(unsigned v) { return __builtin_amdgcn_ballot_w64(v == 0) != 0; }

__global__ __launch_bounds__(64 * NWAVE, 1) void fused(Args a, int rank, unsigned lmap, const char* __restrict__ x,
                                                       const char* __restrict__ W, char* __restrict__ out) {
  __shared__ __attribute__((aligned(1024))) char smem[LDS_TOTAL];
  const int tid = threadIdx.x, lane = tid & 63, w = rfl(tid >> 6);
  const int bid = blockIdx.x;
  const unsigned long long T0 = now();
  const int g = bid & 7, idx = bid >> 3;
  const int h = g & 1, nb = (idx & 7) * 4 + (g >> 1), mb = idx >> 3;
  const int rot = (idx + (idx >> 3)) & 3;
  const int hh = lane >> 5, m = lane & 31;
  char* me = a.ws[7];
  const rsrc_t rX = __builtin_amdgcn_make_buffer_rsrc((void*)x, (short)0, 4096 * 512 * 2, 0x00020000);
  const rsrc_t rMe = __builtin_amdgcn_make_buffer_rsrc((void*)me, (short)0, (int)WS_NEED, 0x00020000);
  const unsigned lbase = (unsigned)(size_t)(__attribute__((address_space(3))) char*)smem;
  const unsigned co = CTR_OFF + (bid * 8 + w) * 4;
  auto timeout = [&](unsigned call) { st1(rMe, TMO_OFF, call); lds_w32(lbase + C_BAD, 0); };
  v4u xv4;
  { const unsigned xv = (lane >> 3) * 1024u + (((lane & 7) ^ (lane >> 4)) << 4);
    xv4 = v4u{xv, (xv ^ 64u) + 7168u, xv + 2 * 7168u, (xv ^ 64u) + 3 * 7168u}; }

  if (w == 0) {
    const int rho = g >= 4 ? idx + (g & 2) * 16 : g == 1 ? 32 + (idx >> 1) : idx;
    const int j = g >= 4 ? rho >> 2 : (rho * 43) >> 7;
    const int li = g >= 4 ? 3 + (rho & 3) : rho - 3 * j;
    const int lk = (lmap >> (3 * li)) & 7, p = (rank + 1 + lk) & 7;
    unsigned msk = (unsigned)(PMASK >> (8 * g)) & 255;
    if (g == 1 && (idx & 1)) msk ^= 255;
    const unsigned ng = __builtin_popcount(msk);
    if ((NOPUSH && !PX) || !msk) { BARRIER(); return; }
    msk |= 256;
    auto nextG = [&]() { const int r = __builtin_ctz(msk) * 16 + j; msk = (msk & (msk - 1)) | 256; return r; };
    int G = nextG(), Gn = nextG(), ph = 0, na = 1;
    auto tA = [&](int gg) { return TBASE + (((gg >> 4) * 32 + (gg & 15)) * PER) / 160; };
    auto ldg = [&](int gg, bool real) {
      const unsigned R = phys(2 * (gg >> 4), (gg >> 2) & 3);
      const unsigned xv = (lane >> 3) * 4096u + (((lane & 7) ^ (2 * (R & 1))) << 4);
      const unsigned o = real ? (unsigned)(p * 512 + (gg & 15) * 32) * 1024u + (R >> 1) * 128u : 0x40000000u;
      asm volatile("buffer_load_dwordx4 v[208:211], %0, %1, %2 offen\n\tbuffer_load_dwordx4 v[212:215], %0, %1, %2 offen offset:1024\n\t"
                   "buffer_load_dwordx4 v[216:219], %0, %1, %2 offen offset:2048\n\tbuffer_load_dwordx4 v[220:223], %0, %1, %2 offen offset:3072"
                   :: "v"(xv), "s"(rX), "s"(o) : "memory", PREGS);
    };
    const rsrc_t rP = __builtin_amdgcn_make_buffer_rsrc((void*)a.ws[li], (short)0, (int)WS_NEED, 0x00020000);
    unsigned cv;
    asm volatile("buffer_load_dword %0, off, %1, %2 sc0 sc1" : "=v"(cv) : "s"(rMe), "s"(co) : "memory");
    int tev = tA(G) - TLD;
    ldg(G, G < 16);
    asm volatile("s_mov_b64 exec, 0xff\n\tbuffer_atomic_add %0, %1, %2, %3 offen sc1\n\ts_mov_b64 exec, -1"
                 :: "v"(1u), "v"(lane * 128u), "s"(rP), "s"(G < 16 ? HINT_OFF + (G >> 2) * 1024 + (6 - lk) * 4 : 0x40000000u) : "memory");
    SB(); BARRIER(); SB();
    if (G < 16) { ph = 1; tev += TLD; }
    if (PPRIO) __builtin_amdgcn_s_setprio(PPRIO);
    unsigned call;
    asm volatile("s_waitcnt vmcnt(5)\n\tv_readfirstlane_b32 %0, %1" : "=s"(call) : "v"(cv) : "memory");
    ++call;
    const unsigned so = SLOT_OFF + ((call + (PX ? 1 : 0)) % 3) * SLOT_BYTES + rank * SRC_BYTES, vo = lane * 16u, tm = tagm(call);
    auto push = [&](int c) {
      if (ph == 1) {
        unsigned ib;
        asm volatile("s_getreg_b32 %0, hwreg(HW_REG_IB_STS)" : "=s"(ib) :: "memory");
        if ((int)((ib & 15) | ((ib >> 18) & 48)) > na) return;
        asm volatile(
                     "s_mov_b32 vcc_lo, 0x55555555\n\t"
                     "s_mov_b32 vcc_hi, 0x55555555\n\t"
                     "v_cndmask_b32_dpp v224, v212, v208, vcc quad_perm:[1,0,3,2] row_mask:0xf bank_mask:0xf\n\t"
                     "v_cndmask_b32_dpp v225, v213, v209, vcc quad_perm:[1,0,3,2] row_mask:0xf bank_mask:0xf\n\t"
                     "v_cndmask_b32_dpp v226, v214, v210, vcc quad_perm:[1,0,3,2] row_mask:0xf bank_mask:0xf\n\t"
                     "v_cndmask_b32_dpp v227, v215, v211, vcc quad_perm:[1,0,3,2] row_mask:0xf bank_mask:0xf\n\t"
                     "v_cndmask_b32_dpp v232, v220, v216, vcc quad_perm:[1,0,3,2] row_mask:0xf bank_mask:0xf\n\t"
                     "v_cndmask_b32_dpp v233, v221, v217, vcc quad_perm:[1,0,3,2] row_mask:0xf bank_mask:0xf\n\t"
                     "v_cndmask_b32_dpp v234, v222, v218, vcc quad_perm:[1,0,3,2] row_mask:0xf bank_mask:0xf\n\t"
                     "v_cndmask_b32_dpp v235, v223, v219, vcc quad_perm:[1,0,3,2] row_mask:0xf bank_mask:0xf\n\t"
                     "s_not_b64 vcc, vcc\n\t"
                     "v_cndmask_b32_dpp v228, v208, v212, vcc quad_perm:[1,0,3,2] row_mask:0xf bank_mask:0xf\n\t"
                     "v_cndmask_b32_dpp v229, v209, v213, vcc quad_perm:[1,0,3,2] row_mask:0xf bank_mask:0xf\n\t"
                     "v_cndmask_b32_dpp v230, v210, v214, vcc quad_perm:[1,0,3,2] row_mask:0xf bank_mask:0xf\n\t"
                     "v_cndmask_b32_dpp v231, v211, v215, vcc quad_perm:[1,0,3,2] row_mask:0xf bank_mask:0xf\n\t"
                     "v_cndmask_b32_dpp v236, v216, v220, vcc quad_perm:[1,0,3,2] row_mask:0xf bank_mask:0xf\n\t"
                     "v_cndmask_b32_dpp v237, v217, v221, vcc quad_perm:[1,0,3,2] row_mask:0xf bank_mask:0xf\n\t"
                     "v_cndmask_b32_dpp v238, v218, v222, vcc quad_perm:[1,0,3,2] row_mask:0xf bank_mask:0xf\n\t"
                     "v_cndmask_b32_dpp v239, v219, v223, vcc quad_perm:[1,0,3,2] row_mask:0xf bank_mask:0xf\n\t"
                     "s_mov_b32 vcc_lo, 0x33333333\n\t"
                     "s_mov_b32 vcc_hi, 0x33333333\n\t"
                     "v_cndmask_b32_dpp v240, v232, v224, vcc quad_perm:[2,3,0,1] row_mask:0xf bank_mask:0xf\n\t"
                     "v_cndmask_b32_dpp v241, v233, v225, vcc quad_perm:[2,3,0,1] row_mask:0xf bank_mask:0xf\n\t"
                     "v_cndmask_b32_dpp v242, v234, v226, vcc quad_perm:[2,3,0,1] row_mask:0xf bank_mask:0xf\n\t"
                     "v_cndmask_b32_dpp v243, v235, v227, vcc quad_perm:[2,3,0,1] row_mask:0xf bank_mask:0xf\n\t"
                     "v_cndmask_b32_dpp v244, v236, v228, vcc quad_perm:[2,3,0,1] row_mask:0xf bank_mask:0xf\n\t"
                     "v_cndmask_b32_dpp v245, v237, v229, vcc quad_perm:[2,3,0,1] row_mask:0xf bank_mask:0xf\n\t"
                     "v_cndmask_b32_dpp v246, v238, v230, vcc quad_perm:[2,3,0,1] row_mask:0xf bank_mask:0xf\n\t"
                     "v_cndmask_b32_dpp v247, v239, v231, vcc quad_perm:[2,3,0,1] row_mask:0xf bank_mask:0xf\n\t"
                     "s_not_b64 vcc, vcc\n\t"
                     "v_cndmask_b32_dpp v248, v224, v232, vcc quad_perm:[2,3,0,1] row_mask:0xf bank_mask:0xf\n\t"
                     "v_cndmask_b32_dpp v249, v225, v233, vcc quad_perm:[2,3,0,1] row_mask:0xf bank_mask:0xf\n\t"
                     "v_cndmask_b32_dpp v250, v226, v234, vcc quad_perm:[2,3,0,1] row_mask:0xf bank_mask:0xf\n\t"
                     "v_cndmask_b32_dpp v251, v227, v235, vcc quad_perm:[2,3,0,1] row_mask:0xf bank_mask:0xf\n\t"
                     "v_cndmask_b32_dpp v252, v228, v236, vcc quad_perm:[2,3,0,1] row_mask:0xf bank_mask:0xf\n\t"
                     "v_cndmask_b32_dpp v253, v229, v237, vcc quad_perm:[2,3,0,1] row_mask:0xf bank_mask:0xf\n\t"
                     "v_cndmask_b32_dpp v254, v230, v238, vcc quad_perm:[2,3,0,1] row_mask:0xf bank_mask:0xf\n\t"
                     "v_cndmask_b32_dpp v255, v231, v239, vcc quad_perm:[2,3,0,1] row_mask:0xf bank_mask:0xf\n\t"
                     "v_xor_b32 v240, %0, v240\n\tv_xor_b32 v244, %0, v244\n\tv_xor_b32 v248, %0, v248\n\tv_xor_b32 v252, %0, v252"
                     :: "s"(tm) : "memory", "vcc", "scc", PREGS);
        ph = 3;
        return;
      }
      if (c < tev) return;
      const unsigned sg = so + (G >> 4) * 65536u + (G & 15) * 1024u;
      if (ph == 3) {
        asm volatile("buffer_store_dwordx4 v[240:243], %0, %1, %2 offen sc0 sc1\n\tbuffer_store_dwordx4 v[244:247], %0, %1, %3 offen sc0 sc1"
                     :: "v"(vo), "s"(rP), "s"(sg), "s"(sg + 16384) : "memory");
        if (Gn < 128) ldg(Gn, true);
        na = 0; ph = 2; tev += PER / 10;
      } else if (ph == 2) {
        asm volatile("buffer_store_dwordx4 v[248:251], %0, %1, %2 offen sc0 sc1\n\tbuffer_store_dwordx4 v[252:255], %0, %1, %3 offen sc0 sc1\n\ts_nop 1"
                     :: "v"(vo), "s"(rP), "s"(sg + 32768), "s"(sg + 49152) : "memory");
        na += 2; ph = 1; G = Gn; Gn = nextG(); tev = tA(G);
      } else { ldg(G, true); na = 0; ph = 1; tev += TLD; }
    };
    #pragma nounroll
    while (G < 128) {
      const int c = (int)(now() - T0);
      push(c);
      if (c > (int)(2 * DEADLINE_TICKS)) break;
      const int dt = tev - c - (ph == 1 ? PW1 : 0);
      if (dt > 45) __builtin_amdgcn_s_sleep(8);
      else if (dt > 12) __builtin_amdgcn_s_sleep(ESL);
      else if (ES0) __builtin_amdgcn_s_sleep(ES0);
    }
    st1(rMe, co, call);
    VMCNT(0);
    if (lane == 0) __hip_atomic_fetch_add((unsigned*)(a.ws[li] + FLAG_OFF + rank * 128), ng, __ATOMIC_RELAXED, __HIP_MEMORY_SCOPE_SYSTEM);
    return;
  }
  if (w == 3) {
    unsigned cv;
    asm volatile("buffer_load_dword %0, off, %1, %2 sc0 sc1" : "=v"(cv) : "s"(rMe), "s"(co) : "memory");
    {
      const unsigned za = lbase + CTRL + lane * 16;
      const v4u ones = {~0u, ~0u, ~0u, ~0u}, pr = {1u << 31, 0u, 1u << 31, 0u};
      asm volatile("ds_write_b128 %0, %2\n\tds_write_b128 %0, %2 offset:1024\n\tds_write_b128 %0, %1 offset:2048\n\tds_write_b128 %0, %1 offset:3072\n\ts_waitcnt lgkmcnt(0)"
                   :: "v"(za), "v"(ones), "v"(pr) : "memory");
    }
    const unsigned hso = HINT_OFF + (mb * 8 + g) * 128;
    auto poll = [&](int np) {
      asm volatile("s_mov_b32 m0, %0\n\ts_mov_b64 exec, 0xff\n\tbuffer_load_dword %1, %2, %3 offen sc0 sc1 lds\n\ts_mov_b64 exec, -1"
                   :: "s"(lbase + C_ARR + np * 32), "v"(lane * 4), "s"(rMe), "s"(hso) : "m0", "memory");
    };
    poll(0);
    SB(); BARRIER(); SB();
    if (SPRIO) __builtin_amdgcn_s_setprio(SPRIO);
    const int d0 = DA + (mb * PER) / 40, lead = d0 + LEAD;
    unsigned eu[2], ao[2], wo[2], la[2];
    int ko[2], nk[2];
    #pragma unroll
    for (int k = 0; k < 2; ++k) {
      const unsigned e = lane + 64 * k, r = (e * 9363u) >> 16, u = e - 7 * r, s = (rank + 1 + u) & 7, R = phys(r, mb);
      eu[k] = e < 112 ? u : 31;
      ko[k] = (r * PER) / 10 + (r >= 2 ? SL2 : 0) - LEAD;
      ao[k] = SLOT_OFF + s * SRC_BYTES + r * 32768u;
      wo[k] = (s * 512u + 64u * (R >> 1) + 16u * (R & 1) + (e < QP ? 8u * h : 0u)) * 8192u;
      nk[k] = NEVER; la[k] = 0;
    }
    unsigned call;
    asm volatile("s_waitcnt vmcnt(1)\n\tv_readfirstlane_b32 %0, %1" : "=s"(call) : "v"(cv) : "memory");
    ++call;
    ao[0] += (call % 3) * SLOT_BYTES; ao[1] += (call % 3) * SLOT_BYTES;
    u64 cm[2] = {0, 0};
    int cnt = 0, nrel = 0;
    auto events = [&](int c) {
      #pragma unroll
      for (int k = 0; k < 2; ++k) {
        u64 ev = bal(nk[k] <= c);
        if (k == 1 && cnt < QP) ev &= (1ull << (QP - 64)) - 1;
        if (ev) {
          const u64 nw = ev & ~cm[k];
          const unsigned pos = __builtin_amdgcn_mbcnt_hi((unsigned)(nw >> 32), __builtin_amdgcn_mbcnt_lo((unsigned)nw, cnt));
          const bool isn = (nw >> lane) & 1;
          if ((ev >> lane) & 1) {
            lds_w32(isn ? lbase + C_WL + pos * 4 : la[k], isn ? wo[k] : ao[k]);
            la[k] = lbase + C_AL + pos * 8;
            if (pos >= 12) ao[k] |= 1u << 31;
            nk[k] = isn ? nk[k] + lead : NEVER;
          }
          cm[k] |= nw;
          cnt += __builtin_popcountll(nw); nrel += __builtin_popcountll(ev & ~nw);
        }
      }
    };
    int tp = PP, np = 1;
    int ti = 0;
    unsigned unk = 0x7f;
    #pragma nounroll
    while (unk) {
      const unsigned f = lds_r32(lbase + C_ARR + lane * 4);
      const unsigned long long el = now() - T0;
      const int c = (int)el;
      if (c >= tp) {
        poll(np);
        if ((lane >> 3) == np) ti = c;
        np = (np + 1) & 7;
        tp = c + PP;
      }
      u64 ok = bal((int)(f - 4 * call) >= 0 && f != ~0u) & (unk * 0x0101010101010101ull);
      if (NOPUSH || el > DEADLINE_TICKS) { ok = unk; ti = NOPUSH ? -2000 : c; }
      while (ok) {
        const int b = __builtin_ctzll(ok) & 56;
        const unsigned bits = (unsigned)(ok >> b) & 0x7f;
        const int ts = rdl(ti, b);
        #pragma unroll
        for (int k = 0; k < 2; ++k) if ((bits >> eu[k]) & 1) nk[k] = min(nk[k], ts + ko[k]);
        ok &= ~(0xffull << b);
        unk &= ~bits;
      }
      events(c);
      if (SSD) __builtin_amdgcn_s_sleep(SSD);
    }
    if (SPRIO2 != SPRIO) __builtin_amdgcn_s_setprio(SPRIO2);
    #pragma nounroll
    while (nrel < 112) {
      const unsigned long long el = now() - T0;
      events((int)el);
      if (el > 2 * DEADLINE_TICKS) break;
      if (nrel < NFINE) __builtin_amdgcn_s_sleep(SSL);
      else __builtin_amdgcn_s_sleep(SSL2);
    }
    if (rfl(__builtin_amdgcn_raw_buffer_load_b32(rMe, 0, DIRTY_OFF + (~call & 1) * 128, 17)) == call) {
      const unsigned zo = SLOT_OFF + ((call + 2) % 3) * SLOT_BYTES + bid * 16384, z = tagm(call + 2);
      #pragma nounroll
      for (int j = 0; j < 16; ++j) __builtin_amdgcn_raw_buffer_store_b128(v4u{z, z, z, z}, rMe, lane * 16, zo + j * 1024, 17);
    }
    st1(rMe, co, call);
    return;
  }
  if (w < 4) {
    const int lw = w - 1;
    if (LPRIO) __builtin_amdgcn_s_setprio(LPRIO);
    const rsrc_t rW = __builtin_amdgcn_make_buffer_rsrc((void*)W, (short)0, 4096 * 4096 * 2, 0x00020000);
    const unsigned wq = lane >> 4;
    const unsigned ch = (lane & 15) ^ (((wq >> 1) << 3) | ((wq & 1) << 1));
    const unsigned wv1 = wq * 8192u + ((ch ^ (8u * h)) << 4) + nb * 256u;
    const unsigned wv2 = (wq + ((ch >> 3) ^ h) * 8) * 8192u + ((ch & 7) << 4) + nb * 256u + h * 128u;
    const v4u wst = {0u, 31744u, 32 * 8192u - 2048u, 36 * 8192u - 3072u};
    v4u wv = wst + wv1;
    const unsigned wb = (unsigned)(rank * 512 + 64 * h + 8 * lw) * 8192u, xb = (unsigned)(rank * 512 + mb * 128 + 32 * lw) * 1024u + 128u * h;
    glds4p<WPOL>(rW, wv, wb + rot * 128 * 8192u, lbase + lw * 4096);
    BARRIER();
    sfor<0, 14>([&](auto IC_) {
      constexpr int i = decltype(IC_)::value, b = (2 * i + 4) >> 3, r = (2 * i + 4) & 7;
      const unsigned sb = (b + rot) & 3;
      if constexpr (r < 4) {
        glds4p<XPOL>(rX, xv4, xb + r * 32768u + sb * 256u, lbase + (16 + 4 * b + r + lw) * 4096);
        gzero(rMe, WS_NEED, lbase + C_XF + (4 * b + r + lw) * 4);
      } else {
        if constexpr (i > 0) glds4p<WPOL>(rW, wv, wb + (sb * 128 + 8 * (r - 4)) * 8192u, lbase + (4 * b + r - 4 + lw) * 4096);
        gzero(rMe, WS_NEED, lbase + C_FL + (4 * b + r - 4 + lw) * 8);
      }
      if constexpr (i >= LD - 1) VMCNT(5 * (LD - 1));
    });
    const unsigned pos = min(lw + 2 * lane, 111), q = (pos * 9363u) >> 16, q5 = q - 5 * ((q * 205u) >> 10);
    const unsigned tla = lbase + ((unsigned)((0x1707100020ull >> (8 * q5)) & 255) + pos - 7 * q) * 4096u;
    const unsigned need = q < 3 ? (q + 1) >> 1 : q - 1;
    int i = 0, spin = 0;
    unsigned fa = lbase + C_FL + (16 + lw) * 8;
    const int isw = (QP + 1 - lw) >> 1;
    #pragma nounroll
    while (i < 56) {
      const unsigned wl = lds_r32(lbase + C_WL + pos * 4), dn = lds_r32(lbase + C_DONE + lane * 4);
      const unsigned seen = __builtin_ctzll(bal(dn == NONE) | (1ull << 63)) >> 2;
      u64 rdy = ~bal(wl != NONE && need <= seen) | (1ull << (i < isw ? isw : 56));
      int e = i + __builtin_ctzll(rdy >> i);
      if (e == i) {
        if ((++spin & 255) == 0 && now() - T0 > 2 * DEADLINE_TICKS) break;
        __builtin_amdgcn_s_sleep(1);
        continue;
      }
      #pragma nounroll
      do {
        glds4p<WPOL>(rW, wv, rdl(wl, i), rdl(tla, i));
        gzero(rMe, WS_NEED, fa);
        VMCNT(5 * (LD - 1));
        if (LSL) __builtin_amdgcn_s_sleep(LSL);
        fa += 16; ++i;
      } while (i < e);
      if (i == isw) wv = wst + wv2;
    }
    VMCNT(0);
    return;
  }
  const int cw = w - 4;
  unsigned cv;
  asm volatile("buffer_load_dword %0, off, %1, %2 sc0 sc1" : "=v"(cv) : "s"(rMe), "s"(co) : "memory");
  glds4p<XPOL>(rX, xv4, (unsigned)(rank * 512 + mb * 128 + 32 * cw) * 1024u + 128u * h + rot * 256u, lbase + (16 + cw) * 4096);
  if (TOUCH == 1) asm volatile("s_mov_b64 exec, 0xfff\n\tbuffer_load_dword v64, %0, %1, %2 offen sc0 sc1\n\ts_mov_b64 exec, -1"
                          :: "v"(lane << 18), "s"(rMe), "s"(SLOT_OFF + cw * (3u << 20)) : "v64", "memory");
  BARRIER();
  if (TOUCH == 2) asm volatile("s_mov_b64 exec, 0xfff\n\tbuffer_load_dword v64, %0, %1, %2 offen sc0 sc1\n\ts_mov_b64 exec, -1"
                          :: "v"(lane << 18), "s"(rMe), "s"(SLOT_OFF + cw * (3u << 20)) : "v64", "memory");
  if (TOUCH == 3) asm volatile("s_mov_b64 exec, 3\n\tbuffer_load_dword v64, %0, %1, %2 offen sc0 sc1\n\ts_mov_b64 exec, -1"
                          :: "v"(lane << 21), "s"(rMe), "s"(SLOT_OFF + cw * (4u << 20) - (2u << 20)) : "v64", "memory");
  if (WPRIO) __builtin_amdgcn_s_setprio(WPRIO);

  f32x16 acc[4];
  #pragma unroll
  for (int i = 0; i < 4; ++i)
    #pragma unroll
    for (int j = 0; j < 16; ++j) acc[i][j] = 0.f;

  unsigned tb[4], tw[4];
  {
    const unsigned q = (lane >> 2) & 3, p = lane & 3, blk = (lane >> 4) & 1;
    unsigned b0 = 256u * (8 * hh + q) + 128u * (q >> 1) + 32u * (q & 1) + 64u * (p & 1) + 16u * blk + 8u * (p >> 1);
    tb[0] = lbase + b0; tb[1] = lbase + (b0 ^ 32u); tb[2] = lbase + (b0 ^ 128u); tb[3] = lbase + (b0 ^ 160u);
  }
  const unsigned xra = lbase + (16 + cw) * 4096 + m * 128 + (((unsigned)(4 * hh) ^ ((m >> 1) & 7)) << 4);
  const unsigned avo = (unsigned)(mb * 4 + cw) * 1024u + (m >> 2) * 128 + hh * 64 + (m & 3) * 16 + h * 16384;
  const unsigned avy = avo ^ 16384u;
  unsigned fad = lbase + C_FL;
  const unsigned xfa = lbase + C_XF + 4 * cw;
  unsigned lad = lbase + C_AL + (lane < 16 ? lane : 120) * 8;
  unsigned dad = lbase + C_DONE + 4 * cw;
  asm volatile("" : "+v"(fad), "+v"(lad), "+v"(dad));
  s16x4 wf[2][8];
  v4u B[14], Y[14];
  unsigned F = 0, L = 1, sL = 0, Lv = 0;
  unsigned nm = (1u << BT(1)) - 1;
  unsigned call, tm, tk;

  auto rdtr = [&](unsigned ad) { return __builtin_amdgcn_ds_read_tr16_b64_v4i16((lds_s16x4_p)(size_t)ad); };
  auto mma = [&](const s16x4 (&f)[8], int i, int ai, const v4u& b) {
    s16x8 aa = __builtin_shufflevector(f[2 * i], f[2 * i + 1], 0, 1, 2, 3, 4, 5, 6, 7);
    acc[ai] = __builtin_amdgcn_mfma_f32_32x32x16_bf16(__builtin_bit_cast(bf16x8, aa), __builtin_bit_cast(bf16x8, b), acc[ai], 0, 0, 0);
  };
  auto ldA = [&](unsigned vo, unsigned so) -> v4u { return __builtin_amdgcn_raw_buffer_load_b128(rMe, vo, so, 17); };
  v4u P[8];
  const unsigned pvo = lane * 16;
  auto ldP = [&](int c0, int c1) {
    const unsigned po = PART_OFF + (bid ^ (SX ? 8 : 1)) * 32768 + cw * 8192;
    #pragma unroll
    for (int c = c0; c < c1; ++c) P[c] = __builtin_amdgcn_raw_buffer_load_b128(rMe, pvo + (c & 3) * 1024, po + (c >> 2) * 4096, 17);
  };
  unsigned sen[2] = {~0u, ~0u};
  auto ldPn = [&](int hf) {
    const unsigned po = PART_OFF + (bid ^ 1) * 32768 + cw * 8192 + hf * 4096;
    asm volatile("v_mov_b32 %4, -1\n\t"
                 "buffer_load_dwordx4 %0, %5, %6, %7 offen sc0 sc1\n\tbuffer_load_dwordx4 %1, %5, %6, %7 offen offset:1024 sc0 sc1\n\t"
                 "buffer_load_dwordx4 %2, %5, %6, %7 offen offset:2048 sc0 sc1\n\tbuffer_load_dwordx4 %3, %5, %6, %7 offen offset:3072 sc0 sc1\n\t"
                 "buffer_load_dword %4, off, %6, %8"
                 : "=&v"(P[4 * hf]), "=&v"(P[4 * hf + 1]), "=&v"(P[4 * hf + 2]), "=&v"(P[4 * hf + 3]), "=&v"(sen[hf])
                 : "v"(pvo), "s"(rMe), "s"(po), "s"(WS_NEED) : "memory");
  };
  auto isbad = [&]() { return rfl(lds_r32(lbase + C_BAD)) == 0; };
  auto gslow = [&](v4u& A, unsigned vo, unsigned so) {
    const unsigned fo = lane == 0 ? TMO_OFF : FLAG_OFF + ((so >> 19) & 7) * 128;
    bool fseen = false;
    #pragma nounroll
    while (true) {
      unsigned fv;
      asm volatile("s_nop 4\n\tbuffer_load_dwordx4 %0, %2, %3, %4 offen sc0 sc1\n\t"
                   "buffer_load_dword %1, %5, %3, 0 offen sc0 sc1\n\ts_waitcnt vmcnt(0)"
                   : "+&v"(A), "=&v"(fv) : "v"(vo), "s"(rMe), "s"(so), "v"(fo) : "memory");
      if (!anyp(tagv(A, tk))) break;
      if (fseen) { lds_w32(lbase + C_DRT, 0); break; }
      fseen = (bal((int)(fv - call * 128) >= 0) & 2) != 0;
      const bool tmo_hit = bal(fv == call) & 1;
      if (tmo_hit || now() - T0 > DEADLINE_TICKS) { timeout(call); break; }
      __builtin_amdgcn_s_sleep(PSLEEP);
    }
  };
  auto fspin = [&](unsigned fa) {
    int spin = 0;
    #pragma nounroll
    while (rfl(lds_r32(fa)) != 0) {
      if ((++spin & 31) == 0 && now() - T0 > 2 * DEADLINE_TICKS) { lds_w32(lbase + C_BAD, 0); break; }
    }
  };
  auto lspin = [&](unsigned fa) {
    int spin = 0;
    unsigned v;
    #pragma nounroll
    while (rfl(v = lds_r32(fa)) == 0) {
      if ((++spin & 31) == 0 && now() - T0 > 2 * DEADLINE_TICKS) { lds_w32(lbase + C_BAD, 0); v = WS_NEED; break; }
    }
    return v;
  };
#define PF_RL(t, l) "v_readlane_b32 s" #t ", v112, " #l "\n\t"
#define PF_LD(a, b, t) "buffer_load_dwordx4 v[" #a ":" #b "], v113, s[68:71], s" #t " offen sc0 sc1\n\t"
#define PF_OUT "+{v[64:67]}"(B[0]), "+{v[68:71]}"(B[1]), "+{v[72:75]}"(B[2]), "+{v[76:79]}"(B[3]), "+{v[80:83]}"(B[4]), "+{v[84:87]}"(B[5]), \
    "+{v[88:91]}"(B[6]), "+{v[92:95]}"(B[7]), "+{v[96:99]}"(B[8]), "+{v[100:103]}"(B[9]), "+{v[104:107]}"(B[10]), "+{v[108:111]}"(B[11]), \
    "+{v112}"(Lv), "+{s60}"(nm)
#define PF_IN "{v113}"(avo), "{s[68:71]}"(rMe)
#define PF_IN2 PF_IN, "{v114}"(lad), "{s61}"((unsigned)T0)
#define PF_CL "s62", "s63", "s64", "s65", "s66", "s67", "scc", "memory"
  auto prefetch = [&](auto PM) {
    constexpr int pm = decltype(PM)::value;
    const int cwl = cw;
    if constexpr (pm == 0) {
      asm volatile("v_cmp_eq_u32 s[62:63], 0, v112\n\ts_and_b32 s62, s62, s60\n\ts_getpc_b64 s[66:67]\n\ts_cbranch_scc0 .Lpf_body"
                   : PF_OUT : PF_IN : PF_CL);
    } else if constexpr (pm == 1) {
      asm volatile("s_getpc_b64 s[66:67]\n\ts_cmp_lg_u32 s60, 0x10000\n\ts_cbranch_scc1 .Lpf_wait\n\ts_setprio " STR(CPRIO) : PF_OUT : PF_IN2 : PF_CL, "vcc");
      if (rdl(Lv, 16) != 0) lds_w32(lbase + C_BAD, 0);
    } else {
      asm volatile(
                   "s_getpc_b64 s[66:67]\n.Lpf_h0:\n\ts_add_u32 s66, s66, .Lpf_h1-.Lpf_h0\n\ts_addc_u32 s67, s67, 0\n\t"
                   "s_cmp_eq_u32 %[cw], 0\n\ts_cbranch_scc1 .Lpf_p0\n\ts_cmp_eq_u32 %[cw], 1\n\ts_cbranch_scc1 .Lpf_p1\n\t"
                   "s_cmp_eq_u32 %[cw], 2\n\ts_cbranch_scc1 .Lpf_p2\n\ts_branch .Lpf_p3\n"
                   ".Lpf_h1:\n\ts_add_u32 s66, s66, .Lpf_h2-.Lpf_h1\n\ts_addc_u32 s67, s67, 0\n\t"
                   "s_cmp_eq_u32 %[cw], 0\n\ts_cbranch_scc1 .Lpf_p4\n\ts_cmp_eq_u32 %[cw], 1\n\ts_cbranch_scc1 .Lpf_p5\n"
                   ".Lpf_h2:\n\t"
                   ".subsection 1\n\t.p2align 6\n"
                   ".Lpf_p0:\n\ts_setpc_b64 s[66:67]\n"
                   ".Lpf_body:\n\ts_cmp_eq_u32 s60, 15\n\ts_cbranch_scc0 .Lpf_1\n\t"
                   PF_RL(62, 0) PF_RL(63, 1) PF_RL(64, 2) PF_RL(65, 3) "s_movk_i32 s60, 0x7f\n\ts_nop 2\n\t"
                   PF_LD(64, 67, 62) PF_LD(68, 71, 63) PF_LD(72, 75, 64) PF_LD(76, 79, 65) "\n.Lpf_p1:\n\ts_setpc_b64 s[66:67]\n"
                   ".Lpf_1:\n\ts_cmpk_eq_u32 s60, 0x7f\n\ts_cbranch_scc0 .Lpf_2\n\t"
                   PF_RL(62, 4) PF_RL(63, 5) PF_RL(64, 6) "s_movk_i32 s60, 0x3ff\n\ts_nop 2\n\t"
                   PF_LD(80, 83, 62) PF_LD(84, 87, 63) PF_LD(88, 91, 64) "\n.Lpf_p2:\n\ts_setpc_b64 s[66:67]\n"
                   ".Lpf_2:\n\ts_cmpk_eq_u32 s60, 0x3ff\n\ts_cbranch_scc0 .Lpf_3\n\t"
                   PF_RL(62, 7) PF_RL(63, 8) PF_RL(64, 9) "s_movk_i32 s60, 0xfff\n\ts_nop 2\n\t"
                   PF_LD(92, 95, 62) PF_LD(96, 99, 63) PF_LD(100, 103, 64) "\n.Lpf_p3:\n\ts_setpc_b64 s[66:67]\n"
                   ".Lpf_3:\n\ts_cmp_eq_u32 s60, s60\n\t"
                   PF_RL(62, 10) PF_RL(63, 11) "s_mov_b32 s60, 0x10000\n\ts_nop 2\n\t"
                   PF_LD(104, 107, 62) PF_LD(108, 111, 63) "\n.Lpf_p4:\n\ts_setpc_b64 s[66:67]\n"
                   ".Lpf_wait:\n\tv_cmp_eq_u32 vcc, 0, v112\n\ts_and_b32 s62, vcc_lo, s60\n\ts_cbranch_scc0 .Lpf_body\n\t"
                   "s_setprio " STR(WPRIO) "\n\tds_read_b32 v112, v114\n\ts_memrealtime vcc\n\ts_waitcnt lgkmcnt(0)\n\t"
                   "s_sub_u32 s62, vcc_lo, s61\n\ts_cmp_lt_u32 s62, %[dl]\n\ts_cbranch_scc1 .Lpf_wait\n\t"
                   "v_mov_b32 v112, %[wn]\n\ts_branch .Lpf_body\n"
                   ".Lpf_p5:\n\ts_setpc_b64 s[66:67]\n\t"
                   ".fill 64, 4, 0xbf800000\n\t.subsection 0"
                   : PF_OUT : PF_IN2, [dl] "n"((unsigned)(2 * DEADLINE_TICKS)), [wn] "n"(WS_NEED), [cw] "s"(cwl) : PF_CL, "vcc");
    }
  };
  constexpr int E0 = 14 - SPL, HS = SPL ? SPL / 2 : 1;
  static_assert(QP >= 82 && QP + 1 + (8 / SPS - 1) * SPP <= 98 + E0 + HS && PLDK2 >= PLDK && SPL <= 12 && SPL % 2 == 0 && PLDK >= 1 && PLDK < E0 + HS, "");
  unsigned bd = 0;
  float cf = -1.44269504088896341f;
  unsigned ew[4], er, eo;
  f32x2 ey, ee;
  v4u eq;
  const unsigned eb = lbase + (14 + (cw & 1) + (cw >> 1) * 16) * 4096;
  auto chkP = [&](int c0, int c1, int c2) {
    auto stale = [&]() {
      unsigned o0 = ~bd;
      #pragma unroll
      for (int c = c0; c < c1; ++c) o0 |= P[c][0] ^ call;
      return __builtin_amdgcn_ballot_w64((o0 & 1) != 0) != 0;
    };
    if (__builtin_expect(stale(), 0)) {
      int it = 0;
      #pragma nounroll
      do {
        bool bad = rfl(bd) == 0;
        if ((++it & 15) == 0) {
          const unsigned tv = rfl(__builtin_amdgcn_raw_buffer_load_b32(rMe, 0, TMO_OFF, 17));
          if (tv == call || now() - T0 > DEADLINE_TICKS) { timeout(call); bad = true; }
        }
        if (bad) { cf = __uint_as_float(0x7FC00000u); bd = 0; break; }
        MEMB();
        ldP(c0, c1); SB(); ldP(c1, c2);
      } while (stale());
    }
  };
  auto fin = [&](int i, int pr, int hf) {
    if (hf == 0) {
      const v4u& q = P[i * 4 + (pr >> 1)];
      ey = f32x2{acc[i][2 * pr], acc[i][2 * pr + 1]} + f32x2{__uint_as_float(q[(2 * pr) & 3]), __uint_as_float(q[((2 * pr) & 3) + 1])};
      const f32x2 z = ey * cf;
      ee = f32x2{__builtin_amdgcn_exp2f(z[0]), __builtin_amdgcn_exp2f(z[1])};
    } else {
      const f32x2 d = ee + 1.f;
      const f32x2 o = ey * f32x2{__builtin_amdgcn_rcpf(d[0]), __builtin_amdgcn_rcpf(d[1])};
      eq[pr & 3] = pk_bf16(o[0], o[1]);
      if ((pr & 3) == 3) *(lds_vv4u_p)(size_t)ew[2 * i + (pr >> 2)] = eq;
    }
  };
  auto step = [&](auto MODE_, auto K_, int rb) {
    constexpr int MODE = decltype(MODE_)::value, K = decltype(K_)::value;
    constexpr int NP = MODE < 2 ? 0 : (MODE == 2 ? 70 : MODE == 4 ? 84 : 98) + K;
    constexpr int PB = (K + 1) & 1;
    constexpr bool LAST = MODE == 3 && K == 14;
    constexpr bool CT = NP - 1 >= QP;
    constexpr bool VY = NP >= QP, PY = NP + 12 >= QP;
    constexpr bool NS = MODE == 3 && K >= E0 && !LAST, CS = MODE == 3 && K - 1 >= E0;
    constexpr int NQ = NS ? (K - E0) / HS : 0, NU = NS ? E0 + 2 * ((K - E0) % HS) : K;
    constexpr int CQ = CS ? (K - 1 - E0) / HS : 0, CU = CS ? E0 + 2 * ((K - 1 - E0) % HS) : 0;
    constexpr bool VAL = MODE != 0 && !LAST && !NOPUSH && NQ == 0;
    constexpr bool FCK = !(MODE == 3 && K >= (SPL ? E0 : 13));
    constexpr bool FALL = SPL && MODE == 3 && K == E0 - 1;
    constexpr bool LCK = MODE == 0 ? K == 15 : !(MODE == 3 && K >= 1);
    constexpr int CB = MODE == 0 ? 12 + ((K + 1) & 1) : (K + 13) % 14;
    constexpr auto wof = [](int u) { return (unsigned)(MODE == 0 ? u : MODE == 3 ? (u < 7 ? u : u + 2) : u % 7) * 4096u; };
    constexpr auto nst = [](int np) { return np > QP && (np - QP - 1) % SPP == 0 && (np - QP - 1) / SPP < 8 / SPS ? SPS : 0; };
    s16x4 (&wc)[8] = wf[PB];
    s16x4 (&wn)[8] = wf[PB ^ 1];
    unsigned t = ~0u, t1 = ~0u;
    bool fail = false;
    auto tr = [&](int j) {
      if constexpr (NS) wn[j] = rdtr(tw[NQ + (j & 2)] + wof(NU + (j >> 2)) + (j & 1) * 1024);
      else if constexpr (!LAST) wn[j] = rdtr(tw[j >> 1] + wof(K) + (j & 1) * 1024);
    };
    #pragma unroll
    for (int i = 0; i < 4; ++i) {
      SB();
      if (i == 0) {
        if constexpr (MODE == 0) B[12 + (K & 1)] = lds_r128v((xra ^ ((K & 3) << 4)) + (K >> 2) * 16384);
        tr(0); tr(1);
        if constexpr (MODE == 0) {
        } else if constexpr (K < 2) {
          B[12 + K] = ldA(avo, sL);
          if constexpr (PY) Y[12 + K] = ldA(avy, sL);
        } else if constexpr (MODE != 3) {
          B[K - 2] = ldA(avo, sL);
          if constexpr (PY) Y[K - 2] = ldA(avy, sL);
        }
      }
      if (i == 1) {
        tr(2); tr(3); tr(4);
        if constexpr (nst(NP) > 0) {
          const unsigned po = PART_OFF + bid * 32768 + cw * 8192;
          const unsigned am = bd & ~1u, ov = (~bd & 0x7FC00000u) | (call & 1);
          #pragma unroll
          for (int c = (NP - QP - 1) / SPP * SPS; c < ((NP - QP - 1) / SPP + 1) * SPS; ++c) {
            v4u q;
            #pragma unroll
            for (int e = 0; e < 4; ++e) q[e] = __float_as_uint(acc[2 + (c >> 2)][4 * (c & 3) + e]);
            q[0] = (q[0] & am) | ov;
            __builtin_amdgcn_raw_buffer_store_b128(q, rMe, pvo + (c & 3) * 1024, po + (c >> 2) * 4096, PAUX);
          }
        }
        if constexpr (VAL) {
          t = tagv(B[NU], tk);
          if constexpr (VY) t1 = tagv(Y[NU], tk);
          if constexpr (NS) t = min3(t, t1, tagv(B[NU + 1], tk));
        }
      }
      if (i == 2) {
        tr(5); tr(6); tr(7);
        if constexpr (VAL && NS) t = min(t, tagv(Y[NU + 1], tk));
        else if constexpr (VAL && VY && LCK) t = min3(t, t1, L);
        else if constexpr (VAL && VY) t = min(t, t1);
        else if constexpr (VAL && LCK) t = min(t, L);
        else if constexpr (LCK) t = L;
        if constexpr (LCK) sL = rfl(L) & ~(1u << 31);
        if constexpr (MODE == 0 && !LCK) fail = bal(F != 0) != 0;
        else if constexpr (FCK) fail = bal((int)t >= (int)F) != 0;
        else if constexpr (VAL) fail = anyp(t);
        if constexpr (MODE == 0 && K >= PF0) prefetch(IC<K == 15>{});
      }
      if (i == 3) {
        if (__builtin_expect(fail, 0)) {
          __builtin_amdgcn_s_setprio(WPRIO);
          if constexpr (FALL) { fspin(fad + 12 * 8); fspin(fad + 13 * 8); }
          else if constexpr (FCK) fspin(fad + (K + 1) * 8);
          if constexpr (MODE == 0 && (K + 1) % 4 == 0 && K < 15) fspin(lbase + C_XF + (K + 1 + cw) * 4);
          if constexpr (LCK) sL = rfl(lspin(fad + (K + 1) * 8 + 4)) & ~(1u << 31);
          if constexpr (VAL) {
            unsigned t2 = tagv(B[NU], tk);
            if constexpr (VY) t2 = min(t2, tagv(Y[NU], tk));
            if constexpr (NS) t2 = min3(t2, tagv(B[NU + 1], tk), tagv(Y[NU + 1], tk));
            const unsigned lv = lds_r32(lad + NU * 8) & ~(1u << 31);
            if (anyp(t2) && !isbad()) {
              constexpr int NE = MODE == 3 ? (K == 0 ? 13 : 14 - NU) : 13;
              auto again = [&](auto J0) {
                sfor<0, NE>([&](auto JC) {
                  constexpr int j = decltype(JC)::value, e = (j + NU) % 14, pp = NP - K + NU + j;
                  const unsigned so = rdl(lv, j);
                  SB();
                  if constexpr (j >= decltype(J0)::value) B[e] = ldA(avo, so);
                  SB();
                  if constexpr (pp >= QP) Y[e] = ldA(avy, so);
                  SB();
                  #pragma unroll
                  for (int c = 0; c < nst(pp - 12); ++c)
                    __builtin_amdgcn_raw_buffer_store_b32(0, rMe, c * 64, 0x40000000, 0);
                });
                SB();
                if constexpr (MODE == 3) sfor<1, K + 1>([&](auto JC) {
                  constexpr int k2 = decltype(JC)::value;
                  #pragma unroll
                  for (int c = 0; c < (k2 >= 2 ? nst(98 + k2) : 0); ++c) __builtin_amdgcn_raw_buffer_store_b32(0, rMe, c * 64, 0x40000000, 0);
                  SB();
                  if constexpr (!PNW && k2 < K && k2 == PLDK) ldP(0, 4);
                  SB();
                  if constexpr (!PNW && k2 < K && k2 == PLDK2) ldP(4, 8);
                  SB();
                });
                SB();
              };
              #pragma nounroll
              for (int z = 0; z < 2; ++z) {
                again(IC<0>{});
                if (z || !anyp(tagv(B[NU], tk))) break;
                gslow(B[NU], avo, rfl(lv));
              }
              if constexpr (VY) {
                if (anyp(tagv(Y[NU], tk))) gslow(Y[NU], avy, rfl(lv));
              }
              if constexpr (NS) {
                if (anyp(tagv(B[NU + 1], tk))) gslow(B[NU + 1], avo, rdl(lv, 1));
                if (anyp(tagv(Y[NU + 1], tk))) gslow(Y[NU + 1], avy, rdl(lv, 1));
              }
            }
          }
          __builtin_amdgcn_s_setprio(CPRIO);
          __builtin_amdgcn_s_waitcnt(PNW && MODE == 3 && K > PLDK ? 0x0070 : 0xC07F);
        }
        SB();
        if constexpr (VAL) {
          B[NU][0] ^= tm;
          if constexpr (VY) Y[NU][0] ^= tm;
          if constexpr (NS) { B[NU + 1][0] ^= tm; Y[NU + 1][0] ^= tm; }
        }
        if constexpr (MODE == 0 ? K >= 14 : MODE != 3) { const v2u fl = lds_r64(fad + (K + 2) * 8); F = fl[0]; L = fl[1]; }
        else if constexpr (SPL && MODE == 3 && K == E0 - 2) F = lds_r32(fad + 12 * 8) | lds_r32(fad + 13 * 8);
        else if constexpr (!(MODE == 3 && K >= (SPL ? E0 - 1 : 12))) F = lds_r32(fad + (K + 2) * 8);
        if constexpr (NP == QP || (MODE == 3 && K == (SPL ? E0 + HS - 1 : 13))) bd = lds_r32(lbase + C_BAD);
        if constexpr (SPL && MODE == 3 && K == E0 + HS) chkP(0, 4, PLDK2 < K ? 8 : 4);
        if constexpr (SPL && LAST) chkP(4, 8, 8);
        if constexpr (MODE == 3 && K == 5) {
          const unsigned rr = lane >> 3, ch = lane & 7;
          #pragma unroll
          for (int c = 0; c < 4; ++c) ew[c] = eb + m * 128 + (((4 * hh + c) ^ (m & 7)) << 4);
          er = eb + rr * 128 + ((ch ^ rr) << 4);
          eo = (unsigned)(mb * 128 + 32 * cw + rr) * 8192u + nb * 256u + h * 128u + ch * 16u;
          asm volatile("" : "+v"(ew[0]), "+v"(ew[1]), "+v"(ew[2]), "+v"(ew[3]), "+v"(er), "+v"(eo));
        }
        if constexpr (MODE == 0 && (K + 2) % 4 == 0 && K < 14) F |= lds_r32(xfa + (K + 2) * 4);
        if constexpr (MODE == 0 && K + 1 >= PF0 && K < 15) Lv = lds_r32(lad);
        if constexpr (MODE == 0 ? (K == 7 || K == 13 || K == 15) : (MODE == 1 ? K % 7 == 6 : MODE == 2 && K == 6))
          asm volatile("s_mov_b64 exec, 1\n\tds_write_b32 %0, %0 offset:%1\n\ts_mov_b64 exec, -1"
                       :: "v"(dad), "n"(MODE == 0 ? (K - 7) / 4 * 16 : 48 + K / 7 * 16) : "memory");
      }
      if constexpr (CQ == 1) {
        constexpr int s0 = (K - 1 - E0 - HS) * 4;
        #pragma unroll
        for (int z = ((s0 + i) * 16 + 4 * HS - 1) / (4 * HS); z < ((s0 + i + 1) * 16 + 4 * HS - 1) / (4 * HS); ++z) if (z >= EOF_) fin(0, (z - EOF_) >> 1, (z - EOF_) & 1);
      }
      SB();
      if constexpr (CS) mma(wc, i, CQ, i & 1 ? Y[CU + (i >> 1)] : B[CU + (i >> 1)]);
      else if constexpr (CT) mma(wc, i, i & 1, i < 2 ? B[CB] : Y[CB]);
      else mma(wc, i, i, B[CB]);
    }
    SB();
  };

  #pragma unroll
  for (int e = 0; e < 12; ++e) asm volatile("" : "=v"(B[e]));
  SB(); prefetch(IC<2>{}); SB();
  asm volatile("s_waitcnt vmcnt(%2)\n\tv_readfirstlane_b32 %0, %1" : "=s"(call) : "v"(cv), "n"(TOUCH ? 1 : 0) : "memory");
  ++call;
  tm = tagm(call); tk = (tm << 2) | (1u << 30);
  B[12] = lds_r128v(xra);
  fspin(fad);
  fspin(fad + 8);
  __builtin_amdgcn_s_setprio(CPRIO);
  #pragma unroll
  for (int i = 0; i < 4; ++i) { tw[i] = tb[i]; wf[0][2 * i] = rdtr(tw[i]); wf[0][2 * i + 1] = rdtr(tw[i] + 1024); }
  F = lds_r32(fad + 16);
  sfor<1, 16>([&](auto NC) { step(IC<0>{}, NC, 0); });
  fad += 16 * 8;
  auto iter = [&](auto MODE_, int r) {
    sfor<0, 14>([&](auto KC) {
      constexpr int k = decltype(KC)::value;
      if constexpr (k % 7 == 0) {
        const unsigned go = ((unsigned)(0x201707100020ull >> (8 * ((r % 5) + k / 7))) & 255) * 4096;
        #pragma unroll
        for (int i = 0; i < 4; ++i) { tw[i] = tb[i] + go; if (TWP) asm volatile("" : "+v"(tw[i])); }
      }
      step(MODE_, KC, r);
    });
    fad += 14 * 8; lad += 14 * 8; dad += 32;
  };
  for (int r = 0; r < 10; r += 2) {
    iter(IC<1>{}, r);
  }
  iter(IC<2>{}, 10);
  iter(IC<4>{}, 12);
  sfor<0, 15>([&](auto KC) {
    constexpr int k = decltype(KC)::value;
    if constexpr (k == 0) {
      #pragma unroll
      for (int i = 0; i < 4; ++i) { tw[i] = tb[i] + gslot(14) * 4096; if (TWP) asm volatile("" : "+v"(tw[i])); }
    }
    step(IC<3>{}, KC, 14);
    if constexpr (k == PLDK) ldP(0, 4);
    SB();
    if constexpr (k == PLDK2) ldP(4, 8);
  });
  if (!SPL) chkP(0, 8, 8);
  {
    #pragma unroll
    for (int z = SPL ? 16 - EOF_ : 0; z < 32; ++z) fin(z >> 4, (z >> 1) & 7, z & 1);
    const rsrc_t rO = __builtin_amdgcn_make_buffer_rsrc((void*)out, (short)0, 512 * 4096 * 2, 0x00020000);
    v4u o[4];
    #pragma unroll
    for (int i = 0; i < 4; ++i) o[i] = lds_r128v(er + i * 1024);
    #pragma unroll
    for (int i = 0; i < 4; ++i) __builtin_amdgcn_raw_buffer_store_b128(o[i], rO, eo, i * 8u * 8192u, OAUX);
    asm volatile("s_nop 1" :: "v"(o[0]), "v"(o[1]), "v"(o[2]), "v"(o[3]) : "memory");
  }
  st1(rMe, co, call);
  if (anyz(lds_r32(lbase + C_BAD + (lane & 1) * 4))) st1(rMe, DIRTY_OFF + (call & 1) * 128, call + 1);
}

extern "C" void kernel_launch(int rank, int nranks, ncclComm_t comm,
                              hipStream_t stream,
                              void* const* d_in, const int* in_sizes, int n_in,
                              void* d_out, int out_size,
                              const int* in_dtypes, int out_dtype,
                              void* const* d_out_all,
                              void* const* d_ws_all, size_t ws_bytes) {
  if (nranks != NR || ws_bytes < WS_NEED + WS_GUARD || (unsigned)rank >= (unsigned)NR) {
    fprintf(stderr, "unsupported configuration\n");
    return;
  }
  Args a;
  a.ws[7] = (char*)d_ws_all[rank];
  unsigned lmap = 0;
  for (int l = 0, n0 = 0, n1 = 3; l < 7; ++l) {
    const bool same = ((TOPO >> rank) & 1) == ((TOPO >> ((rank + 1 + l) & 7)) & 1);
    const int li = same ? n0++ : n1++;
    lmap |= (unsigned)l << (3 * li);
    a.ws[li] = (char*)d_ws_all[(rank + 1 + l) & 7];
  }
  fused<<<256, 64 * NWAVE, 0, stream>>>(a, rank, lmap, (const char*)d_in[0], (const char*)d_in[1], (char*)d_out);
  HIP_CHECK(hipGetLastError());
}
```
